```python
import numpy as np

kernel_name = "gpu_gemm_ar_m4096_k4096_n16384_bf16_silu"


ABSMAX_RTOL = 0.008

COMPUTE_DTYPE = "bfloat16"
EPILOGUE = "silu"
IN_QUANT = None
OUT_QUANT = None
IN_DTYPES = {"x": "bfloat16", "w_mat": "bfloat16"}
WORLD_SIZE = 8

def _to_bf16(x):
    u = np.ascontiguousarray(x, dtype=np.float32).view(np.uint32)
    return ((u + 0x7FFF + ((u >> 16) & 1)) & 0xFFFF0000).view(np.float32)

def setup_inputs():

    x = _to_bf16(np.random.randn(4096, 4096).astype(np.float32))
    w_mat = _to_bf16((np.random.randn(4096, 16384) / np.sqrt(4096)).astype(np.float32))
    return {"x": x, "w_mat": w_mat}

def _epilogue(y):
    return (y / (1.0 + np.exp(-np.clip(y, -60.0, 60.0)))).astype(np.float32)

def reference(x, w_mat):
    y = (x @ w_mat).astype(np.float32)
    return _epilogue(y)
```

```cpp
#include <hip/hip_runtime.h>
#include <rccl/rccl.h>
#include <cstdio>

#define HIP_CHECK(expr) do { hipError_t _e = (expr); if (_e != hipSuccess) { \
    fprintf(stderr, "HIP error %s at %s:%d\n", hipGetErrorString(_e), __FILE__, __LINE__); \
    return; } } while (0)

namespace {

#define LAS __attribute__((address_space(3)))
typedef short bf16x8 __attribute__((ext_vector_type(8)));
typedef float f32x4 __attribute__((ext_vector_type(4)));
typedef unsigned u32x4 __attribute__((ext_vector_type(4)));
typedef unsigned u32x2 __attribute__((ext_vector_type(2)));

constexpr int NRANK = 8;
constexpr int M_ = 4096, N_ = 16384, KSH = 512;
#define CFG_NO 4
#define CFG_CPL 8
#define CFG_U 4
constexpr int NO = CFG_NO;
constexpr int NREP = 64 - 8 * NO;
constexpr int NCB = NO + NREP;
constexpr int NWG = 256;
constexpr int NCOMMSLOT = 64;
constexpr int CPL = CFG_CPL;
constexpr int NCOMM = 7 * CPL;
constexpr int NG = NWG - NCOMMSLOT;
constexpr int GPX = NG / 8;
constexpr int CBW = GPX / 2;
constexpr int NW = CPL * 8;
constexpr int U = CFG_U;
constexpr int UPB = 64 / U;
static_assert(NCOMM <= NCOMMSLOT, "comm slots");
static_assert(CBW >= NO, "own blocks are in the first round");
constexpr int NCB1 = 8 * NO + (CBW - NO);
constexpr int IMG_STAGE = 32 + 2 * NCB1;
constexpr int IMG_REST = (64 - NCB1) * 8;
constexpr int NIMG = 4 * IMG_STAGE + IMG_REST;
static_assert(NIMG == 640, "image count");
constexpr int NBLK_IN = 64 + 4 * NCB;
constexpr int NBLK1 = 4 * (16 + CBW);

constexpr size_t MiB = 1ull << 20;
constexpr size_t OFF_SEQ = 0;
constexpr size_t OFF_TMO = 32768;
constexpr size_t OFF_EXP = 36864;
constexpr size_t OFF_LOCX = 45056;
constexpr size_t OFF_LOCW = 49152;
constexpr size_t OFF_IMPX = 57344;
constexpr size_t OFF_IMPW = 65536;
constexpr size_t OFF_YF = 98304;
constexpr size_t OFF_DBG = 262144;
constexpr size_t OFF_BUF = 1 * MiB;
constexpr size_t BUF_BYTES = 160 * MiB;
constexpr size_t W_OFF = 32 * MiB;
constexpr size_t OFF_Y = OFF_BUF + 2 * BUF_BYTES;
constexpr size_t Y_BYTES = 16 * MiB;
constexpr size_t WS_NEED = OFF_Y + 2 * Y_BYTES;
constexpr unsigned long long DEADLINE = 100000000ull;

constexpr int LDS_BYTES = 131072 + 64;

struct Params {
    const unsigned short* x;
    const unsigned short* w;
    unsigned short* out;
    char* ws[NRANK];
    int rank;
};

#define RLX_AGENT __ATOMIC_RELAXED, __HIP_MEMORY_SCOPE_AGENT
#define RLX_SYS __ATOMIC_RELAXED, __HIP_MEMORY_SCOPE_SYSTEM
#define WAIT_V(n) asm volatile("s_waitcnt vmcnt(" #n ")" ::: "memory")
#define WAIT_L(n) asm volatile("s_waitcnt lgkmcnt(" #n ")" ::: "memory")
#define BAR __builtin_amdgcn_s_barrier()
#define SCHED __builtin_amdgcn_sched_barrier(0)

__device__ __forceinline__ unsigned* wptr(char* base, size_t off) { return (unsigned*)(base + off); }
__device__ __forceinline__ __amdgpu_buffer_rsrc_t mkrsrc(const void* p, unsigned bytes) {
    return __builtin_amdgcn_make_buffer_rsrc((void*)p, 0, bytes, 0x00020000);
}
__device__ __forceinline__ int gcb(int rank, int i) { return i < NO ? rank * NO + i : 8 * NO + (i - NO); }

__device__ __forceinline__ bool wave_wait(unsigned* f, unsigned need, unsigned* tmo, unsigned call) {
    unsigned v = __builtin_amdgcn_readfirstlane(__hip_atomic_load(f, RLX_SYS));
    if ((int)(v - need) >= 0) return true;
    const unsigned long long t0 = __builtin_amdgcn_s_memrealtime();
    for (;;) {
        __builtin_amdgcn_s_sleep(2);
        v = __builtin_amdgcn_readfirstlane(__hip_atomic_load(f, RLX_SYS));
        if ((int)(v - need) >= 0) return true;
        if ((unsigned)__builtin_amdgcn_readfirstlane(__hip_atomic_load(tmo, RLX_AGENT)) == call ||
            __builtin_amdgcn_s_memrealtime() - t0 > DEADLINE) {
            __hip_atomic_store(tmo, call, RLX_AGENT);
            return false;
        }
    }
}

__device__ __forceinline__ unsigned pack_bf16(float lo, float hi) {
    unsigned r; asm("v_cvt_pk_bf16_f32 %0, %1, %2" : "=v"(r) : "v"(lo), "v"(hi)); return r;
}
__device__ __forceinline__ float silu(float y) {
    const float c = __builtin_fminf(__builtin_fmaxf(y, -60.0f), 60.0f);
    const float e = __builtin_amdgcn_exp2f(c * -1.44269504088896f);
    return y * __builtin_amdgcn_rcpf(1.0f + e);
}

#define STAMP(slot) do {} while (0)

__global__ void __launch_bounds__(512, 2) fused_gemm_allgather_silu(const Params p) {
    extern __shared__ __attribute__((aligned(16))) unsigned char shm_[];
    LAS unsigned char* lds = (LAS unsigned char*)shm_;
    LAS unsigned* ctl = (LAS unsigned*)(lds + 131072);
    const int tid = threadIdx.x, lane = tid & 63;
    const int wid = __builtin_amdgcn_readfirstlane(tid >> 6);
    const int rank = p.rank;
    const int b = blockIdx.x;
    char* const myws = p.ws[rank];
    unsigned* const tmo = wptr(myws, OFF_TMO);
    const bool is_gemm = b >= NCOMMSLOT;
    if (wid == 0) STAMP(0);

    if (tid == 0) ctl[0] = __hip_atomic_fetch_add(wptr(myws, OFF_SEQ + (size_t)b * 128), 1u, RLX_AGENT) + 1u;

    unsigned call;
    {
        const int nimg = is_gemm ? 3 : 1;
        u32x4 v[3][4];
        int img[3];
#pragma unroll
        for (int q = 0; q < 3; ++q) {
            img[q] = q == 0 ? b : NWG + (q - 1) * NG + (b - NCOMMSLOT);
            if (q < nimg) {
                int i = img[q], isx, blk, kt;
                if (i < 4 * IMG_STAGE) { const int j = i / IMG_STAGE, r = i % IMG_STAGE; isx = r < 32; const int r2 = isx ? r : r - 32; blk = r2 >> 1; kt = 2 * j + (r2 & 1); }
                else { const int r = i - 4 * IMG_STAGE; isx = 0; blk = NCB1 + (r >> 3); kt = r & 7; }
                if (isx) {
#pragma unroll
                    for (int e = 0; e < 4; ++e) {
                        const int bi = wid * 4 + e;
                        const int row = blk * 256 + (bi >> 1) * 16 + (lane & 15), kc = kt * 64 + (bi & 1) * 32 + (lane >> 4) * 8;
                        v[q][e] = *(const u32x4*)(p.x + (size_t)row * KSH + kc);
                    }
                } else {
                    const unsigned short* src = p.w + (size_t)(kt * 64 + wid * 8) * N_ + blk * 256 + lane * 4;
#pragma unroll
                    for (int e = 0; e < 4; ++e) {
                        const u32x2 r0 = *(const u32x2*)(src + (size_t)(2 * e) * N_), r1 = *(const u32x2*)(src + (size_t)(2 * e + 1) * N_);
                        v[q][e] = (u32x4){r0.x, r0.y, r1.x, r1.y};
                    }
                }
            }
        }
        __syncthreads();
        call = __builtin_amdgcn_readfirstlane(ctl[0]);
        char* const bufp = myws + OFF_BUF + (size_t)(call & 1) * BUF_BYTES;
        const __amdgpu_buffer_rsrc_t rx = mkrsrc(bufp + (size_t)rank * 4 * MiB, 4u << 20);
        const __amdgpu_buffer_rsrc_t rw = mkrsrc(bufp + W_OFF + (size_t)rank * 16 * MiB, 16u << 20);
#pragma unroll
        for (int q = 0; q < 3; ++q) {
            if (q < nimg) {
                int i = img[q], isx, blk, kt, stage;
                if (i < 4 * IMG_STAGE) { const int j = i / IMG_STAGE, r = i % IMG_STAGE; isx = r < 32; const int r2 = isx ? r : r - 32; blk = r2 >> 1; kt = 2 * j + (r2 & 1); stage = j; }
                else { const int r = i - 4 * IMG_STAGE; isx = 0; blk = NCB1 + (r >> 3); kt = r & 7; stage = 4; }
                if (isx) {
                    const unsigned base = (unsigned)(((kt >> 1) * 16 + blk) * 65536 + (kt & 1) * 32768);
#pragma unroll
                    for (int e = 0; e < 4; ++e)
                        __builtin_amdgcn_raw_buffer_store_b128(v[q][e], rx, base + (wid * 4 + e) * 1024 + lane * 16, 0, 17);
                } else {
                    const unsigned base = (unsigned)(blk * 262144 + kt * 32768);
                    const unsigned lo = base + (unsigned)((lane >> 4) * 8192 + (wid >> 2) * 1024 + ((wid & 3) * 16 + (lane & 15)) * 16);
                    u32x4 o[4];
#pragma unroll
                    for (int e = 0; e < 4; ++e) {
                        o[0][e] = __builtin_amdgcn_perm(v[q][e].z, v[q][e].x, 0x05040100u);
                        o[1][e] = __builtin_amdgcn_perm(v[q][e].z, v[q][e].x, 0x07060302u);
                        o[2][e] = __builtin_amdgcn_perm(v[q][e].w, v[q][e].y, 0x05040100u);
                        o[3][e] = __builtin_amdgcn_perm(v[q][e].w, v[q][e].y, 0x07060302u);
                    }
#pragma unroll
                    for (int e = 0; e < 4; ++e)
                        __builtin_amdgcn_raw_buffer_store_b128(o[e], rw, lo + e * 2048, 0, 17);
                }
                WAIT_V(0);
                __syncthreads();
                if (wid == 0) {
                    if (lane < 7) {
                        const unsigned peer = (unsigned)(rank + 1 + lane) & 7u;
                        __hip_atomic_fetch_add(wptr(p.ws[peer], OFF_EXP + (size_t)(rank * 8 + stage) * 128), 1u, RLX_SYS);
                    } else if (lane == 7) {
                        __hip_atomic_fetch_add(wptr(myws, (isx ? OFF_LOCX : OFF_LOCW) + (size_t)blk * 128), 1u, RLX_AGENT);
                    }
                }
            }
        }
    }
    if (wid == 0) STAMP(1);
    const unsigned par = call & 1u;

    if (!is_gemm) {
        if (b >= NCOMM) return;
        const int link = b % 7;
        const int wi = (b / 7) * 8 + wid;
        const int src = (rank + 1 + link) & 7;
        char* const rws = p.ws[src];
        const size_t boff = OFF_BUF + (size_t)par * BUF_BYTES;
        const __amdgpu_buffer_rsrc_t rsx = mkrsrc(rws + boff + (size_t)src * 4 * MiB, 4u << 20);
        const __amdgpu_buffer_rsrc_t rsw = mkrsrc(rws + boff + W_OFF + (size_t)src * 16 * MiB, 16u << 20);
        const __amdgpu_buffer_rsrc_t rdx = mkrsrc(myws + boff + (size_t)src * 4 * MiB, 4u << 20);
        const __amdgpu_buffer_rsrc_t rdw = mkrsrc(myws + boff + W_OFF + (size_t)src * 16 * MiB, 16u << 20);
        int known = -1;
        bool ok = true;
        unsigned* pend = nullptr;
        for (int u = wi; u < NBLK_IN * UPB; u += NW) {
            const int blk = u / UPB, sub = u % UPB;
            int isx, j, idx;
            if (blk < NBLK1) { j = blk / (16 + CBW); const int r = blk % (16 + CBW); isx = r < 16; idx = isx ? r : gcb(rank, r - 16); }
            else {
                const int r2 = blk - NBLK1, g = r2 / (4 * CBW), rr = r2 % (4 * CBW);
                const int left = NCB - CBW * (g + 1), n = left < CBW ? left : CBW;
                j = rr / n; isx = 0; idx = gcb(rank, CBW * (g + 1) + rr % n);
            }
            const int stage = (isx || idx < NCB1) ? j : 4;
            while (known < stage) {
                ++known;
                ok = wave_wait(wptr(myws, OFF_EXP + (size_t)(src * 8 + known) * 128), call * (unsigned)(known < 4 ? IMG_STAGE : IMG_REST), tmo, call) && ok;
            }
            const unsigned off = (unsigned)((isx ? (j * 16 + idx) : (idx * 4 + j)) * 65536 + sub * (U * 1024) + lane * 16);
            u32x4 v[U];
            if (isx) {
#pragma unroll
                for (int e = 0; e < U; ++e) v[e] = __builtin_amdgcn_raw_buffer_load_b128(rsx, off + e * 1024, 0, 17);
            } else {
#pragma unroll
                for (int e = 0; e < U; ++e) v[e] = __builtin_amdgcn_raw_buffer_load_b128(rsw, off + e * 1024, 0, 17);
            }
            if (pend) {
                asm volatile("s_waitcnt vmcnt(%0)" :: "i"(U) : "memory");
                if (lane == 0) __hip_atomic_fetch_add(pend, 1u, RLX_AGENT);
            }
            if (isx) {
#pragma unroll
                for (int e = 0; e < U; ++e) __builtin_amdgcn_raw_buffer_store_b128(v[e], rdx, off + e * 1024, 0, 16);
            } else {
#pragma unroll
                for (int e = 0; e < U; ++e) __builtin_amdgcn_raw_buffer_store_b128(v[e], rdw, off + e * 1024, 0, 16);
            }
            asm volatile("" ::: "memory");
            pend = wptr(myws, isx ? OFF_IMPX + (size_t)(j * 16 + idx) * 128 : OFF_IMPW + (size_t)(idx * 4 + j) * 128);
        }
        if (pend) {
            WAIT_V(0);
            if (lane == 0) __hip_atomic_fetch_add(pend, 1u, RLX_AGENT);
        }
        if (wid == 0) STAMP(2);
        const __amdgpu_buffer_rsrc_t rsy = mkrsrc(rws + OFF_Y + (size_t)par * Y_BYTES, (unsigned)(NO * 16 * 131072));
        constexpr int UPT = 128 / U;
        for (int u = wi; u < NO * 16 * UPT; u += NW) {
            const int t = u / UPT, sub = u % UPT;
            ok = wave_wait(wptr(myws, OFF_YF + (size_t)(src * 128 + t) * 128), call * 8u, tmo, call) && ok;
            const int ci = t >> 4, rb = t & 15;
            const unsigned off = (unsigned)(t * 131072 + sub * (U * 1024) + lane * 16);
            u32x4 v[U];
#pragma unroll
            for (int e = 0; e < U; ++e) v[e] = __builtin_amdgcn_raw_buffer_load_b128(rsy, off + e * 1024, 0, 17);
            if (!ok || (unsigned)__builtin_amdgcn_readfirstlane(__hip_atomic_load(tmo, RLX_AGENT)) == call) {
                ok = false;
#pragma unroll
                for (int e = 0; e < U; ++e) v[e] = (u32x4){0x7fc07fc0u, 0x7fc07fc0u, 0x7fc07fc0u, 0x7fc07fc0u};
            }
#pragma unroll
            for (int e = 0; e < U; ++e) {
                const int kb = sub * U + e;
                const int w = kb >> 4, ix = kb & 15;
                const int bj = ix >> 3, nf = (ix >> 2) & 1, ai = (ix >> 1) & 1, hf = ix & 1;
                const int m = rb * 256 + bj * 128 + (w & 3) * 32 + nf * 16 + (lane & 15);
                const int n = (src * NO + ci) * 256 + ai * 128 + (w >> 2) * 64 + (lane >> 4) * 16 + hf * 8;
                *(u32x4*)(p.out + (size_t)m * N_ + n) = v[e];
            }
        }
        if (wid == 0) STAMP(3);
        return;
    }

    {
        const int g = b - NCOMMSLOT, xcd = g & 7, slot = g >> 3;
        const int wr = wid >> 2, wc = wid & 3;
        const char* const bufx = myws + OFF_BUF + (size_t)par * BUF_BYTES;
        const char* const bufw = bufx + W_OFF;
        const unsigned voff = (unsigned)tid * 16u;
        const unsigned ldsw = (unsigned)wid * 1024u;
        const int aoff = wr * 8192 + lane * 16, boff = wc * 4096 + lane * 16;
        const int ntile = (2 * NCB - slot + GPX - 1) / GPX;
        if (ntile <= 0) return;
#define SA(bb, h) (((bb) * 2 + (h)) * 16384)
#define SB(bb, h) ((4 + (bb) * 2 + (h)) * 16384)
#define STAGE(bufoff, gbase) do { _Pragma("unroll") for (int _i = 0; _i < 2; ++_i) \
        __builtin_amdgcn_global_load_lds((const unsigned*)((gbase) + voff + _i * 8192), (LAS unsigned*)(lds + (bufoff) + ldsw + _i * 8192), 16, 0, 0); } while (0)
#define LDA(dst, bb, h) do { _Pragma("unroll") for (int m = 0; m < 4; ++m) _Pragma("unroll") for (int k = 0; k < 2; ++k) dst[m][k] = *(const LAS bf16x8*)(lds + SA(bb, h) + aoff + m * 2048 + k * 1024); } while (0)
#define LDB(dst, bb, h) do { _Pragma("unroll") for (int n = 0; n < 2; ++n) _Pragma("unroll") for (int k = 0; k < 2; ++k) dst[n][k] = *(const LAS bf16x8*)(lds + SB(bb, h) + boff + n * 2048 + k * 1024); } while (0)
#define MMA(ai, bj, At, Bt) do { __builtin_amdgcn_s_setprio(1); _Pragma("unroll") for (int m = 0; m < 4; ++m) _Pragma("unroll") for (int n = 0; n < 2; ++n) _Pragma("unroll") for (int k = 0; k < 2; ++k) \
        acc[ai][bj][m][n] = __builtin_amdgcn_mfma_f32_16x16x32_bf16(At[m][k], Bt[n][k], acc[ai][bj][m][n], 0, 0, 0); __builtin_amdgcn_s_setprio(0); } while (0)

        auto pair_src = [&](int pp) { return pp < 4 ? rank : ((rank + 1 + (pp - 4) % 7) & 7); };
        auto pair_j = [&](int pp) { return pp < 4 ? pp : (pp - 4) / 7; };
        auto wbase = [&](int cb, int pp) { return bufw + (size_t)pair_src(pp) * 16 * MiB + (size_t)(cb * 4 + pair_j(pp)) * 65536; };
        auto xbase = [&](int rb, int pp) { return bufx + (size_t)pair_src(pp) * 4 * MiB + (size_t)(pair_j(pp) * 16 + rb) * 65536; };
        auto ready = [&](int rb, int cb, int seg) -> int {
            if (wid == 0) {
                const int l = lane & 7, side = (lane >> 3) & 1;
                const bool act = lane < 16 && l < 5;
                size_t o;
                if (side == 0) o = l == 0 ? OFF_LOCX + (size_t)rb * 128 : OFF_IMPX + (size_t)(((l - 1) & 3) * 16 + rb) * 128;
                else o = l == 0 ? OFF_LOCW + (size_t)cb * 128 : OFF_IMPW + (size_t)(cb * 4 + ((l - 1) & 3)) * 128;
                const unsigned need = call * (l == 0 ? 8u : (unsigned)(7 * UPB));
                unsigned* f = wptr(myws, o);
                const unsigned long long t0 = __builtin_amdgcn_s_memrealtime();
                int nrdy;
                for (;;) {
                    const unsigned v = act ? __hip_atomic_load(f, RLX_AGENT) : need;
                    const unsigned long long bal = __ballot((int)(v - need) >= 0);
                    const unsigned both = (unsigned)(bal & (bal >> 8)) & 0x1fu;
                    nrdy = __builtin_ctz(~(both | ((1u << seg) - 1u)));
                    if (nrdy > seg) break;
                    if ((unsigned)__builtin_amdgcn_readfirstlane(__hip_atomic_load(tmo, RLX_AGENT)) == call ||
                        __builtin_amdgcn_s_memrealtime() - t0 > DEADLINE) {
                        __hip_atomic_store(tmo, call, RLX_AGENT);
                        nrdy = 5; break;
                    }
                    __builtin_amdgcn_s_sleep(4);
                }
                __builtin_amdgcn_fence(__ATOMIC_ACQUIRE, "agent");
                if (lane == 0) ctl[1] = (unsigned)nrdy;
                asm volatile("s_waitcnt vmcnt(0) lgkmcnt(0)" ::: "memory");
            }
            asm volatile("" ::: "memory"); BAR; asm volatile("" ::: "memory");
            const int nr = __builtin_amdgcn_readfirstlane(*(volatile LAS unsigned*)&ctl[1]);
            WAIT_L(0);
            return nr >= 5 ? 32 : 4 + 7 * (nr - 1);
        };

        f32x4 acc[2][2][4][2];
#pragma unroll
        for (int a = 0; a < 2; ++a)
#pragma unroll
            for (int c = 0; c < 2; ++c)
#pragma unroll
                for (int m = 0; m < 4; ++m)
#pragma unroll
                    for (int n = 0; n < 2; ++n) acc[a][c][m][n] = (f32x4){0.f, 0.f, 0.f, 0.f};
        bf16x8 At[4][2], B0[2][2], B1[2][2];
        int ti = slot;
        int ci = ti >> 1, rb = 2 * xcd + (ti & 1), cb = gcb(rank, ci);
        int lim = ready(rb, cb, 0);
        const char* cA = wbase(cb, 0); const char* cB = xbase(rb, 0);
        STAGE(SB(0, 0), cB); STAGE(SA(0, 0), cA); STAGE(SB(0, 1), cB + 16384); STAGE(SA(0, 1), cA + 16384);
        if (wr == 1) BAR;
        WAIT_V(4); BAR;
        STAGE(SB(1, 0), cB + 32768); STAGE(SA(1, 0), cA + 32768); STAGE(SB(1, 1), cB + 49152);
        WAIT_V(6); BAR;
        if (wid == 0) STAMP(2);
        for (int it = 0; it < ntile; ++it) {
            const bool has_next = it + 1 < ntile;
            const int nti = has_next ? ti + GPX : ti;
            const int nci = nti >> 1, nrb = 2 * xcd + (nti & 1), ncb = gcb(rank, nci);
            for (int pp = 0; pp < 32; ++pp) {
                const bool last = pp == 31;
                const char* a2; const char* b2;
                if (!last) {
                    if (pp + 1 >= lim) lim = ready(rb, cb, 1 + (pp + 1 - 4) / 7);
                    a2 = wbase(cb, pp + 1); b2 = xbase(rb, pp + 1);
                } else {
                    if (has_next) lim = ready(nrb, ncb, 0);
                    a2 = wbase(ncb, 0); b2 = xbase(nrb, 0);
                }
                LDB(B0, 0, 0); SCHED; LDA(At, 0, 0); STAGE(SA(1, 1), cA + 49152);
                WAIT_L(8); BAR; WAIT_L(0); MMA(0, 0, At, B0); BAR; SCHED;
                LDB(B1, 0, 1); STAGE(SB(0, 0), b2);
                BAR; WAIT_L(0); MMA(0, 1, At, B1); BAR;
                LDA(At, 0, 1); STAGE(SA(0, 0), a2);
                BAR; WAIT_L(0); MMA(1, 0, At, B0); BAR; SCHED;
                STAGE(SB(0, 1), b2 + 16384);
                WAIT_V(6); BAR; MMA(1, 1, At, B1); BAR;
                LDB(B0, 1, 0); SCHED; LDA(At, 1, 0); STAGE(SA(0, 1), a2 + 16384);
                WAIT_L(8); BAR; WAIT_L(0); MMA(0, 0, At, B0); BAR; SCHED;
                LDB(B1, 1, 1); STAGE(SB(1, 0), b2 + 32768);
                BAR; WAIT_L(0); MMA(0, 1, At, B1); BAR;
                LDA(At, 1, 1); STAGE(SA(1, 0), a2 + 32768);
                BAR; WAIT_L(0); MMA(1, 0, At, B0); BAR; SCHED;
                STAGE(SB(1, 1), b2 + 49152);
                WAIT_V(6); BAR; MMA(1, 1, At, B1); BAR;
                cA = a2; cB = b2;
            }
            if (wid == 0) STAMP(4 + 2 * it);
            {
                int tz = tid; asm volatile("" : "+v"(tz));
                const int ln = tz & 63, fr = ln & 15, fq = ln >> 4;
                const bool bad = (unsigned)__builtin_amdgcn_readfirstlane(__hip_atomic_load(tmo, RLX_AGENT)) == call;
                const bool own = ci < NO;
                const __amdgpu_buffer_rsrc_t ry = mkrsrc(myws + OFF_Y + (size_t)par * Y_BYTES, own ? (unsigned)(NO * 16 * 131072) : 0u);
                const unsigned yoff = (unsigned)((ci * 16 + rb) * 131072 + wid * 16384 + ln * 16);
                unsigned short* orow = p.out + (size_t)(rb * 256 + wc * 32 + fr) * N_ + cb * 256 + wr * 64 + fq * 16;
#pragma unroll
                for (int bj = 0; bj < 2; ++bj)
#pragma unroll
                    for (int nf = 0; nf < 2; ++nf)
#pragma unroll
                        for (int ai = 0; ai < 2; ++ai)
#pragma unroll
                            for (int hf = 0; hf < 2; ++hf) {
                                u32x4 o;
#pragma unroll
                                for (int d = 0; d < 4; ++d) {
                                    const int j = hf * 2 + (d >> 1), m = (d & 1) * 2;
                                    o[d] = pack_bf16(silu(acc[ai][bj][m][nf][j]), silu(acc[ai][bj][m + 1][nf][j]));
                                }
                                if (bad) o = (u32x4){0x7fc07fc0u, 0x7fc07fc0u, 0x7fc07fc0u, 0x7fc07fc0u};
                                *(u32x4*)(orow + (size_t)(bj * 128 + nf * 16) * N_ + ai * 128 + hf * 8) = o;
                                __builtin_amdgcn_raw_buffer_store_b128(o, ry, yoff + (((bj * 2 + nf) * 2 + ai) * 2 + hf) * 1024, 0, 17);
                            }
                if (own) {
                    WAIT_V(0);
                    if (ln < 7) {
                        const unsigned peer = (unsigned)(rank + 1 + ln) & 7u;
                        __hip_atomic_fetch_add(wptr(p.ws[peer], OFF_YF + (size_t)(rank * 128 + ci * 16 + rb) * 128), 1u, RLX_SYS);
                    }
                }
            }
            if (wid == 0) STAMP(5 + 2 * it);
#pragma unroll
            for (int a = 0; a < 2; ++a)
#pragma unroll
                for (int c = 0; c < 2; ++c)
#pragma unroll
                    for (int m = 0; m < 4; ++m)
#pragma unroll
                        for (int n = 0; n < 2; ++n) acc[a][c][m][n] = (f32x4){0.f, 0.f, 0.f, 0.f};
            ti = nti; ci = nci; rb = nrb; cb = ncb;
        }
        WAIT_V(0);
        if (wr == 0) BAR;
    }
}

}

extern "C" void kernel_launch(int rank, int nranks, ncclComm_t comm,
                              hipStream_t stream,
                              void* const* d_in, const int* in_sizes, int n_in,
                              void* d_out, int out_size,
                              const int* in_dtypes, int out_dtype,
                              void* const* d_out_all,
                              void* const* d_ws_all, size_t ws_bytes) {
    (void)comm; (void)in_sizes; (void)in_dtypes; (void)out_dtype; (void)d_out_all; (void)out_size;
    if (nranks != NRANK || n_in < 2 || ws_bytes < WS_NEED || (unsigned)rank >= (unsigned)NRANK) {
        fprintf(stderr, "unexpected configuration\n");
        return;
    }
    HIP_CHECK(hipFuncSetAttribute((const void*)fused_gemm_allgather_silu, hipFuncAttributeMaxDynamicSharedMemorySize, LDS_BYTES));
    Params p;
    p.x = (const unsigned short*)d_in[0];
    p.w = (const unsigned short*)d_in[1];
    p.out = (unsigned short*)d_out;
    for (int i = 0; i < NRANK; ++i) p.ws[i] = (char*)d_ws_all[i];
    p.rank = rank;
    hipLaunchKernelGGL(fused_gemm_allgather_silu, dim3(NWG), dim3(512), LDS_BYTES, stream, p);
    HIP_CHECK(hipGetLastError());
}
```

```cpp
#include <hip/hip_runtime.h>
#include <rccl/rccl.h>
#include <cstdio>

#define HIP_CHECK(expr) do { hipError_t _e = (expr); if (_e != hipSuccess) { \
    fprintf(stderr, "HIP error %s at %s:%d\n", hipGetErrorString(_e), __FILE__, __LINE__); \
    return; } } while (0)

namespace {

#define LAS __attribute__((address_space(3)))
typedef short bf16x8 __attribute__((ext_vector_type(8)));
typedef float f32x4 __attribute__((ext_vector_type(4)));
typedef unsigned u32x4 __attribute__((ext_vector_type(4)));
typedef unsigned u32x2 __attribute__((ext_vector_type(2)));

constexpr int NRANK = 8;
constexpr int M_ = 4096, N_ = 16384, KSH = 512;
#define CFG_NO 4
#define CFG_CPL 4
#define CFG_U 8
#define CFG_D 4
#define CFG_SLEEP 16
constexpr int NO = CFG_NO;
constexpr int NREP = 64 - 8 * NO;
constexpr int NCB = NO + NREP;
constexpr int NWG = 256;
constexpr int NCOMMSLOT = 64;
constexpr int CPL = CFG_CPL;
constexpr int NCOMM = 7 * CPL;
constexpr int NG = NWG - NCOMMSLOT;
constexpr int CBW = NG / 16;
constexpr int NW = CPL * 8;
constexpr int U = CFG_U;
constexpr int D = CFG_D;
constexpr int UPB = 64 / U;
constexpr int UPT = 128 / U;
static_assert(NCOMM <= NCOMMSLOT, "comm slots");
static_assert(CBW >= NO && CBW == 12, "own blocks are in the first round");
static_assert(D < U && D <= 4 && (U == 4 || U == 8), "pipeline shape");
constexpr int NCB1 = 8 * NO + (CBW - NO);
constexpr int NTASK = 320 - 16;
constexpr unsigned CNT_S0 = 64, CNT_S1 = 2 * NCB1 + 16, CNT_S2 = 2 * NCB1 + 32, CNT_S3 = (64 - NCB1) * 4, CNT_Y = 128;
static_assert(NTASK <= 2 * NG, "tasks");
constexpr int NBLK_IN = 64 + 4 * NCB;
constexpr int NBLK1 = 4 * (16 + CBW);
constexpr int NU_IN = NBLK_IN * UPB, NU = NU_IN + NO * 16 * UPT;

constexpr size_t MiB = 1ull << 20;
constexpr size_t OFF_SEQ = 0;
constexpr size_t OFF_TMO = 32768;
constexpr size_t OFF_TK = 33792;
constexpr size_t OFF_EXP = 98304;
constexpr size_t OFF_LOCX = 45056;
constexpr size_t OFF_LOCW = 49152;
constexpr size_t OFF_IMPX = 57344;
constexpr size_t OFF_IMPW = 65536;
constexpr size_t CTL_BYTES = 131072;
constexpr size_t OFF_DBG = 262144;
constexpr size_t OFF_BUF = 1 * MiB;
constexpr size_t BUF_BYTES = 160 * MiB;
constexpr size_t W_OFF = 32 * MiB;
constexpr size_t OFF_Y = OFF_BUF + 2 * BUF_BYTES;
constexpr size_t Y_BYTES = 16 * MiB;
constexpr size_t WS_NEED = OFF_Y + 2 * Y_BYTES;
constexpr unsigned long long DEADLINE = 100000000ull;

constexpr int LDS_BYTES = 131072 + 64;

struct Params {
    const unsigned short* x;
    const unsigned short* w;
    unsigned short* out;
    char* ws[NRANK];
    int rank;
};

#define RLX_AGENT __ATOMIC_RELAXED, __HIP_MEMORY_SCOPE_AGENT
#define RLX_SYS __ATOMIC_RELAXED, __HIP_MEMORY_SCOPE_SYSTEM
#define WAIT_V(n) asm volatile("s_waitcnt vmcnt(" #n ")" ::: "memory")
#define WAIT_L(n) asm volatile("s_waitcnt lgkmcnt(" #n ")" ::: "memory")
#define BAR __builtin_amdgcn_s_barrier()
#define SCHED __builtin_amdgcn_sched_barrier(0)

__device__ __forceinline__ unsigned* wptr(char* base, size_t off) { return (unsigned*)(base + off); }
__device__ __forceinline__ __amdgpu_buffer_rsrc_t mkrsrc(const void* p, unsigned bytes) {
    return __builtin_amdgcn_make_buffer_rsrc((void*)p, 0, bytes, 0x00020000);
}
typedef int i32x4 __attribute__((ext_vector_type(4)));
__device__ __forceinline__ i32x4 mkdesc(const void* p, unsigned bytes) {
    const unsigned long long a = (unsigned long long)p;
    i32x4 d;
    d.x = __builtin_amdgcn_readfirstlane((int)(unsigned)a);
    d.y = __builtin_amdgcn_readfirstlane((int)((unsigned)(a >> 32) & 0xffffu));
    d.z = __builtin_amdgcn_readfirstlane((int)bytes);
    d.w = 0x00020000;
    return d;
}
template <int OFF> __device__ __forceinline__ void ld_sys(u32x4& v, const i32x4& d, unsigned voff) {
    asm volatile("buffer_load_dwordx4 %0, %1, %2, 0 offen offset:%3 sc0 sc1" : "=v"(v) : "v"(voff), "s"(d), "i"(OFF) : "memory");
}
__device__ __forceinline__ void st_wt(const u32x4& v, const i32x4& d, unsigned voff) {
    asm volatile("buffer_store_dwordx4 %0, %1, %2, 0 offen sc1\n\ts_nop 1" :: "v"(v), "v"(voff), "s"(d) : "memory");
}
__device__ __forceinline__ void add_noret(const i32x4& d, unsigned voff, unsigned one) {
    asm volatile("buffer_atomic_add %0, %1, %2, 0 offen" :: "v"(one), "v"(voff), "s"(d) : "memory");
}
__device__ __forceinline__ void add_ret(unsigned& r, const i32x4& d, unsigned voff) {
    asm volatile("buffer_atomic_add %0, %1, %2, 0 offen sc0" : "+v"(r) : "v"(voff), "s"(d) : "memory");
}
template <int N> __device__ __forceinline__ void wait_for(u32x4& v) { asm volatile("s_waitcnt vmcnt(%1)" : "+v"(v) : "i"(N) : "memory"); }
constexpr int younger_ops(int e) {
    int n = 0;
    const int s0 = e - D;
    for (int s = s0; s <= e; ++s) {
        const int m = (s + U) % U;
        if (s > s0) n += (m == U - D ? 1 : 0) + 1;
        if (s < e) n += 1 + (m == D ? 1 : 0);
    }
    return n;
}
template <int E> struct IC { static constexpr int value = E; };
__device__ __forceinline__ int gcb(int rank, int i) { return i < NO ? rank * NO + i : 8 * NO + (i - NO); }

__device__ __forceinline__ bool wave_wait(unsigned* f, unsigned need, unsigned* tmo, unsigned call) {
    unsigned v = __builtin_amdgcn_readfirstlane(__hip_atomic_load(f, RLX_SYS));
    if ((int)(v - need) >= 0) return true;
    const unsigned long long t0 = __builtin_amdgcn_s_memrealtime();
    for (;;) {
        __builtin_amdgcn_s_sleep(2);
        v = __builtin_amdgcn_readfirstlane(__hip_atomic_load(f, RLX_SYS));
        if ((int)(v - need) >= 0) return true;
        if ((unsigned)__builtin_amdgcn_readfirstlane(__hip_atomic_load(tmo, RLX_AGENT)) == call ||
            __builtin_amdgcn_s_memrealtime() - t0 > DEADLINE) {
            __hip_atomic_store(tmo, call, RLX_AGENT);
            return false;
        }
    }
}

__device__ __forceinline__ unsigned pack_bf16(float lo, float hi) {
    unsigned r; asm("v_cvt_pk_bf16_f32 %0, %1, %2" : "=v"(r) : "v"(lo), "v"(hi)); return r;
}
__device__ __forceinline__ float silu(float y) {
    const float c = __builtin_fminf(__builtin_fmaxf(y, -60.0f), 60.0f);
    const float e = __builtin_amdgcn_exp2f(c * -1.44269504088896f);
    return y * __builtin_amdgcn_rcpf(1.0f + e);
}

#define STAMP(slot) do {} while (0)

__global__ void __launch_bounds__(512, 2) fused_gemm_allgather_silu(const Params p) {
    extern __shared__ __attribute__((aligned(16))) unsigned char shm_[];
    LAS unsigned char* lds = (LAS unsigned char*)shm_;
    LAS unsigned* ctl = (LAS unsigned*)(lds + 131072);
    const int tid = threadIdx.x, lane = tid & 63;
    const int wid = __builtin_amdgcn_readfirstlane(tid >> 6);
    const int rank = p.rank;
    const int b = blockIdx.x;
    char* const myws = p.ws[rank];
    unsigned* const tmo = wptr(myws, OFF_TMO);
    const bool is_gemm = b >= NCOMMSLOT;
    if (wid == 0) STAMP(0);

    if (tid == 0) ctl[0] = __hip_atomic_fetch_add(wptr(myws, OFF_SEQ + (size_t)b * 128), 1u, RLX_AGENT) + 1u;

    unsigned call;
    auto publish = [&](int stage, bool isx, int blk, unsigned cnt) {
        if (wid == 0) {
            if (lane < 7) {
                const unsigned peer = (unsigned)(rank + 1 + lane) & 7u;
                __hip_atomic_fetch_add(wptr(p.ws[peer], OFF_EXP + (size_t)(rank * 16 + stage) * 128), 1u, RLX_SYS);
            } else if (lane == 7) {
                __hip_atomic_fetch_add(wptr(myws, (isx ? OFF_LOCX : OFF_LOCW) + (size_t)blk * 128), cnt, RLX_AGENT);
            }
        }
    };
    if (!is_gemm) {
        const int rb = b >> 2;
        u32x4 v[2];
#pragma unroll
        for (int e = 0; e < 2; ++e) {
            const int pi = (b & 3) * 16 + wid * 2 + e;
            const int row = rb * 256 + ((pi >> 1) & 15) * 16 + (lane & 15), kc = (pi >> 5) * 64 + (pi & 1) * 32 + (lane >> 4) * 8;
            v[e] = *(const u32x4*)(p.x + (size_t)row * KSH + kc);
        }
        __syncthreads();
        call = __builtin_amdgcn_readfirstlane(ctl[0]);
        const __amdgpu_buffer_rsrc_t rx = mkrsrc(myws + OFF_BUF + (size_t)(call & 1) * BUF_BYTES + (size_t)rank * 4 * MiB, 4u << 20);
#pragma unroll
        for (int e = 0; e < 2; ++e)
            __builtin_amdgcn_raw_buffer_store_b128(v[e], rx, (unsigned)(rb * 65536 + ((b & 3) * 16 + wid * 2 + e) * 1024 + lane * 16), 0, 17);
        WAIT_V(0);
        __syncthreads();
        publish(0, true, rb, 1u);
    } else {
        __syncthreads();
        call = __builtin_amdgcn_readfirstlane(ctl[0]);
        char* const bufp = myws + OFF_BUF + (size_t)(call & 1) * BUF_BYTES;
        for (int t = b - NCOMMSLOT; t < NTASK; t += NG) {
            int isx = 0, blk, j, stage;
            if (t < NCB1) { blk = t; j = 0; stage = 1; }
            else {
                const int t1 = t - NCB1;
                if (t1 < 3 * (16 + NCB1)) { j = 1 + t1 / (16 + NCB1); const int r = t1 % (16 + NCB1); isx = r < 16; blk = isx ? r : r - 16; stage = j == 1 ? 1 : 2; }
                else { const int r = t1 - 3 * (16 + NCB1); blk = NCB1 + (r >> 2); j = r & 3; stage = 3; }
            }
            u32x4 v[8];
            if (isx) {
                const __amdgpu_buffer_rsrc_t rx = mkrsrc(bufp + (size_t)rank * 4 * MiB, 4u << 20);
#pragma unroll
                for (int e = 0; e < 8; ++e) {
                    const int pi = wid * 8 + e;
                    const int row = blk * 256 + ((pi >> 1) & 15) * 16 + (lane & 15), kc = j * 128 + (pi >> 5) * 64 + (pi & 1) * 32 + (lane >> 4) * 8;
                    v[e] = *(const u32x4*)(p.x + (size_t)row * KSH + kc);
                }
#pragma unroll
                for (int e = 0; e < 8; ++e)
                    __builtin_amdgcn_raw_buffer_store_b128(v[e], rx, (unsigned)((j * 16 + blk) * 65536 + (wid * 8 + e) * 1024 + lane * 16), 0, 17);
            } else {
                const __amdgpu_buffer_rsrc_t rw = mkrsrc(bufp + W_OFF + (size_t)rank * 16 * MiB, 16u << 20);
                const int o = wid * 2 + (lane >> 5);
                const unsigned short* src = p.w + (size_t)(j * 128 + o * 8) * N_ + blk * 256 + (lane & 31) * 8;
#pragma unroll
                for (int e = 0; e < 8; ++e) v[e] = *(const u32x4*)(src + (size_t)e * N_);
                const unsigned lo = (unsigned)((blk * 4 + j) * 65536 + (o >> 3) * 32768 + ((lane >> 4) & 1) * 8192 + ((o >> 2) & 1) * 1024 + ((o & 3) * 16 + (lane & 15)) * 16);
#pragma unroll
                for (int f = 0; f < 8; ++f) {
                    u32x4 ov;
#pragma unroll
                    for (int q = 0; q < 4; ++q)
                        ov[q] = __builtin_amdgcn_perm(v[2 * q + 1][f >> 1], v[2 * q][f >> 1], (f & 1) ? 0x07060302u : 0x05040100u);
                    __builtin_amdgcn_raw_buffer_store_b128(ov, rw, lo + (f >> 2) * 16384 + (f & 3) * 2048, 0, 17);
                }
            }
            WAIT_V(0);
            __syncthreads();
            publish(stage, isx, blk, 4u);
        }
    }
    if (wid == 0) STAMP(1);
    const unsigned par = call & 1u;

    if (!is_gemm) {
        if (b >= NCOMM) return;
        const int link = b % 7;
        const int src = (rank + 1 + link) & 7;
        const unsigned xoff = (unsigned)(OFF_BUF + (size_t)par * BUF_BYTES + (size_t)src * 4 * MiB);
        const unsigned woff = (unsigned)(OFF_BUF + (size_t)par * BUF_BYTES + W_OFF + (size_t)src * 16 * MiB);
        const unsigned yoff = (unsigned)(OFF_Y + (size_t)par * Y_BYTES);
        const i32x4 dsrc = mkdesc(p.ws[src], (unsigned)WS_NEED);
        const i32x4 dctl = mkdesc(myws, (unsigned)CTL_BYTES);
        const unsigned only0 = lane == 0 ? 0u : 0x80000000u;
        const unsigned lane_in = (unsigned)lane * 16u, lane_out = (unsigned)((lane >> 4) * 4 * N_ + (lane & 15) * 8) * 2u;
        const unsigned tbase = (call - 1u) * (unsigned)(NU + 2 * NW);
        const unsigned tkoff = only0 | (unsigned)(OFF_TK + link * 128);
        const unsigned one = 1u;
        int known = -1;
        bool ok = true;
        auto resolve = [&](unsigned u, unsigned& so, unsigned& dof, unsigned& sig) {
            int stage;
            if (u < (unsigned)NU_IN) {
                const int blk = u / UPB, sub = u % UPB;
                int isx, j, idx;
                if (blk < NBLK1) {
                    j = blk / (16 + CBW); const int r = blk % (16 + CBW); isx = r < 16;
                    idx = isx ? ((r + 2 * link) & 15) : gcb(rank, (r - 16 + 2 * link) % CBW);
                    stage = (isx && j == 0) ? 0 : (j < 2 ? 1 : 2);
                } else {
                    const int r2 = blk - NBLK1, g = r2 / (4 * CBW), rr = r2 % (4 * CBW);
                    const int left = NCB - CBW * (g + 1), n = left < CBW ? left : CBW;
                    j = rr / n; isx = 0; idx = gcb(rank, CBW * (g + 1) + (rr % n + 2 * link) % n);
                    stage = 3;
                }
                so = (isx ? xoff + (unsigned)(j * 16 + idx) * 65536u : woff + (unsigned)(idx * 4 + j) * 65536u) + (unsigned)sub * (U * 1024);
                dof = so;
                sig = isx ? (unsigned)OFF_IMPX + (unsigned)(j * 16 + idx) * 128u : (unsigned)OFF_IMPW + (unsigned)(idx * 4 + j) * 128u;
            } else if (u < (unsigned)NU) {
                const int uy = (int)u - NU_IN, t = uy / UPT, kb = (uy % UPT) * U;
                const int ci = t >> 4, rb = t & 15, w = kb >> 4, ix = kb & 15;
                const int m = rb * 256 + (ix >> 3) * 128 + (w & 3) * 32 + ((ix >> 2) & 1) * 16;
                const int n = (src * NO + ci) * 256 + (w >> 2) * 128;
                so = yoff + (unsigned)uy * (U * 1024);
                dof = (unsigned)(m * N_ + n) * 2u;
                sig = 0x80000000u;
                stage = 4 + ci;
            } else { so = 0x80000000u; dof = 0x80000000u; sig = 0x80000000u; stage = -1; }
            while (known < stage) {
                ++known;
                const unsigned cnt = known == 0 ? CNT_S0 : known == 1 ? CNT_S1 : known == 2 ? CNT_S2 : known == 3 ? CNT_S3 : CNT_Y;
                ok = wave_wait(wptr(myws, OFF_EXP + (size_t)(src * 16 + known) * 128), call * cnt, tmo, call) && ok;
                ok = ok && (unsigned)__builtin_amdgcn_readfirstlane(__hip_atomic_load(tmo, RLX_AGENT)) != call;
            }
        };
        unsigned t0v = 1u, tn = 1u;
        add_ret(t0v, dctl, tkoff);
        add_ret(tn, dctl, tkoff);
        asm volatile("s_waitcnt vmcnt(0)" : "+v"(t0v), "+v"(tn) :: "memory");
        unsigned u = (unsigned)__builtin_amdgcn_readfirstlane(t0v) - tbase;
        unsigned so, dof, sig, psig = 0x80000000u;
        resolve(u, so, dof, sig);
        if (wid == 0) STAMP(16);
        u32x4 v[U];
        {
            const unsigned sv = so + lane_in;
            auto pro = [&](auto ec) {
                constexpr int e = decltype(ec)::value;
                if constexpr (e < D) {
                    ld_sys<e * 1024>(v[e], dsrc, sv);
                    add_noret(dctl, 0x80000000u, one);
                    if constexpr (U - D + e == D) add_noret(dctl, 0x80000000u, one);
                }
            };
            pro(IC<0>{}); pro(IC<1>{}); pro(IC<2>{}); pro(IC<3>{});
        }
        while (u < (unsigned)NU) {
            const bool isout = u >= (unsigned)NU_IN;
            const i32x4 ddst = mkdesc(isout ? (const void*)p.out : (const void*)myws, isout ? (unsigned)((size_t)M_ * N_ * 2) : (unsigned)WS_NEED);
            const unsigned dv = dof + (isout ? lane_out : lane_in);
            const unsigned s0 = isout ? (unsigned)N_ * 2u : 1024u, s1 = isout ? (unsigned)N_ * 32u : 4096u;
            const unsigned sv = so + lane_in, sv4 = sv + 4096u;
            const bool okc = ok;
            unsigned un = 0, nso = 0, ndof = 0, nsig = 0, nsv = 0;
            auto step = [&](auto ec) {
                constexpr int e = decltype(ec)::value;
                if constexpr (e < U) {
                    if constexpr (e + D < U) {
                        ld_sys<((e + D) & 3) * 1024>(v[e + D], dsrc, e + D < 4 ? sv : sv4);
                    } else {
                        if constexpr (e + D == U) {
                            asm volatile("" : "+v"(tn));
                            un = (unsigned)__builtin_amdgcn_readfirstlane(tn) - tbase;
                            tn = 1u;
                            add_ret(tn, dctl, tkoff);
                            resolve(un, nso, ndof, nsig);
                            nsv = nso + lane_in;
                        }
                        ld_sys<(e + D - U) * 1024>(v[e + D - U], dsrc, nsv);
                    }
                    wait_for<younger_ops(e)>(v[e]);
                    if (!okc) v[e] = (u32x4){0x7fc07fc0u, 0x7fc07fc0u, 0x7fc07fc0u, 0x7fc07fc0u};
                    st_wt(v[e], ddst, dv + (e & 3) * s0 + (e >> 2) * s1);
                    if constexpr (e == D) add_noret(dctl, only0 | psig, one);
                }
            };
            step(IC<0>{}); step(IC<1>{}); step(IC<2>{}); step(IC<3>{});
            step(IC<4>{}); step(IC<5>{}); step(IC<6>{}); step(IC<7>{});
            psig = sig;
            u = un; so = nso; dof = ndof; sig = nsig;
        }
        WAIT_V(0);
        add_noret(dctl, only0 | psig, one);
        if (wid == 0) STAMP(3);
        return;
    }

    {
        const int g = b - NCOMMSLOT;
        const int cl = 6 * ((g >> 2) & 1) + (g >> 5), rb = 4 * (g & 3) + ((g >> 3) & 3);
        const int wr = wid >> 2, wc = wid & 3;
        const char* const bufx = myws + OFF_BUF + (size_t)par * BUF_BYTES;
        const char* const bufw = bufx + W_OFF;
        const unsigned voff = (unsigned)tid * 16u;
        const unsigned ldsw = (unsigned)wid * 1024u;
        const int aoff = wr * 8192 + lane * 16, boff = wc * 4096 + lane * 16;
        const int ntile = (NCB - cl + CBW - 1) / CBW;
        if (ntile <= 0) return;
#define SA(bb, h) (((bb) * 2 + (h)) * 16384)
#define SB(bb, h) ((4 + (bb) * 2 + (h)) * 16384)
#define STAGE(bufoff, gbase) do { _Pragma("unroll") for (int _i = 0; _i < 2; ++_i) \
        __builtin_amdgcn_global_load_lds((const unsigned*)((gbase) + voff + _i * 8192), (LAS unsigned*)(lds + (bufoff) + ldsw + _i * 8192), 16, 0, 0); } while (0)
#define LDA(dst, bb, h) do { _Pragma("unroll") for (int m = 0; m < 4; ++m) _Pragma("unroll") for (int k = 0; k < 2; ++k) dst[m][k] = *(const LAS bf16x8*)(lds + SA(bb, h) + aoff + m * 2048 + k * 1024); } while (0)
#define LDB(dst, bb, h) do { _Pragma("unroll") for (int n = 0; n < 2; ++n) _Pragma("unroll") for (int k = 0; k < 2; ++k) dst[n][k] = *(const LAS bf16x8*)(lds + SB(bb, h) + boff + n * 2048 + k * 1024); } while (0)
#define MMA(ai, bj, At, Bt) do { __builtin_amdgcn_s_setprio(1); _Pragma("unroll") for (int m = 0; m < 4; ++m) _Pragma("unroll") for (int n = 0; n < 2; ++n) _Pragma("unroll") for (int k = 0; k < 2; ++k) \
        acc[ai][bj][m][n] = __builtin_amdgcn_mfma_f32_16x16x32_bf16(Bt[n][k], At[m][k], acc[ai][bj][m][n], 0, 0, 0); __builtin_amdgcn_s_setprio(0); } while (0)

        auto pair_src = [&](int pp) { return pp < 4 ? rank : ((rank + 1 + (pp - 4) % 7) & 7); };
        auto pair_j = [&](int pp) { return pp < 4 ? pp : (pp - 4) / 7; };
        auto wbase = [&](int cb, int pp) { return bufw + (size_t)pair_src(pp) * 16 * MiB + (size_t)(cb * 4 + pair_j(pp)) * 65536; };
        auto xbase = [&](int rb, int pp) { return bufx + (size_t)pair_src(pp) * 4 * MiB + (size_t)(pair_j(pp) * 16 + rb) * 65536; };
        auto ready = [&](int rb, int cb, int seg) -> int {
            if (wid == 0) {
                const int l = lane & 7, side = (lane >> 3) & 1;
                const bool act = lane < 16 && l < 5;
                size_t o;
                if (side == 0) o = l == 0 ? OFF_LOCX + (size_t)rb * 128 : OFF_IMPX + (size_t)(((l - 1) & 3) * 16 + rb) * 128;
                else o = l == 0 ? OFF_LOCW + (size_t)cb * 128 : OFF_IMPW + (size_t)(cb * 4 + ((l - 1) & 3)) * 128;
                const unsigned need = call * (l == 0 ? 16u : (unsigned)(7 * UPB));
                unsigned* f = wptr(myws, o);
                const unsigned long long t0 = __builtin_amdgcn_s_memrealtime();
                int nrdy;
                for (;;) {
                    const unsigned v = act ? __hip_atomic_load(f, RLX_AGENT) : need;
                    const unsigned long long bal = __ballot((int)(v - need) >= 0);
                    const unsigned both = (unsigned)(bal & (bal >> 8)) & 0x1fu;
                    nrdy = __builtin_ctz(~(both | ((1u << seg) - 1u)));
                    if (nrdy > seg) break;
                    if ((unsigned)__builtin_amdgcn_readfirstlane(__hip_atomic_load(tmo, RLX_AGENT)) == call ||
                        __builtin_amdgcn_s_memrealtime() - t0 > DEADLINE) {
                        __hip_atomic_store(tmo, call, RLX_AGENT);
                        nrdy = 5; break;
                    }
                    __builtin_amdgcn_s_sleep(CFG_SLEEP);
                }
                __builtin_amdgcn_fence(__ATOMIC_ACQUIRE, "agent");
                if (lane == 0) ctl[1] = (unsigned)nrdy;
                asm volatile("s_waitcnt vmcnt(0) lgkmcnt(0)" ::: "memory");
            }
            asm volatile("" ::: "memory"); BAR; asm volatile("" ::: "memory");
            const int nr = __builtin_amdgcn_readfirstlane(*(volatile LAS unsigned*)&ctl[1]);
            WAIT_L(0);
            return nr >= 5 ? 32 : 4 + 7 * (nr - 1);
        };

        f32x4 acc[2][2][4][2];
#pragma unroll
        for (int a = 0; a < 2; ++a)
#pragma unroll
            for (int c = 0; c < 2; ++c)
#pragma unroll
                for (int m = 0; m < 4; ++m)
#pragma unroll
                    for (int n = 0; n < 2; ++n) acc[a][c][m][n] = (f32x4){0.f, 0.f, 0.f, 0.f};
        bf16x8 At[4][2], B0[2][2], B1[2][2];
        int ci = cl, cb = gcb(rank, ci);
        int lim = ready(rb, cb, 0);
        const char* cA = wbase(cb, 0); const char* cB = xbase(rb, 0);
        STAGE(SB(0, 0), cB); STAGE(SA(0, 0), cA); STAGE(SB(0, 1), cB + 16384); STAGE(SA(0, 1), cA + 16384);
        if (wr == 1) BAR;
        WAIT_V(4); BAR;
        STAGE(SB(1, 0), cB + 32768); STAGE(SA(1, 0), cA + 32768); STAGE(SB(1, 1), cB + 49152);
        WAIT_V(6); BAR;
        if (wid == 0) STAMP(2);
        for (int it = 0; it < ntile; ++it) {
            const bool has_next = it + 1 < ntile;
            const int nci = has_next ? ci + CBW : ci, nrb = rb, ncb = gcb(rank, nci);
            for (int pp = 0; pp < 32; ++pp) {
                const bool last = pp == 31;
                const char* a2; const char* b2;
                if (!last) {
                    if (pp + 1 >= lim) lim = ready(rb, cb, 1 + (pp + 1 - 4) / 7);
                    a2 = wbase(cb, pp + 1); b2 = xbase(rb, pp + 1);
                } else {
                    if (has_next) lim = ready(nrb, ncb, 0);
                    a2 = wbase(ncb, 0); b2 = xbase(nrb, 0);
                }
                LDB(B0, 0, 0); SCHED; LDA(At, 0, 0); STAGE(SA(1, 1), cA + 49152);
                WAIT_L(8); BAR; WAIT_L(0); MMA(0, 0, At, B0); BAR; SCHED;
                LDB(B1, 0, 1); STAGE(SB(0, 0), b2);
                BAR; WAIT_L(0); MMA(0, 1, At, B1); BAR;
                LDA(At, 0, 1); STAGE(SA(0, 0), a2);
                BAR; WAIT_L(0); MMA(1, 0, At, B0); BAR; SCHED;
                STAGE(SB(0, 1), b2 + 16384);
                WAIT_V(6); BAR; MMA(1, 1, At, B1); BAR;
                LDB(B0, 1, 0); SCHED; LDA(At, 1, 0); STAGE(SA(0, 1), a2 + 16384);
                WAIT_L(8); BAR; WAIT_L(0); MMA(0, 0, At, B0); BAR; SCHED;
                LDB(B1, 1, 1); STAGE(SB(1, 0), b2 + 32768);
                BAR; WAIT_L(0); MMA(0, 1, At, B1); BAR;
                LDA(At, 1, 1); STAGE(SA(1, 0), a2 + 32768);
                BAR; WAIT_L(0); MMA(1, 0, At, B0); BAR; SCHED;
                STAGE(SB(1, 1), b2 + 49152);
                WAIT_V(6); BAR; MMA(1, 1, At, B1); BAR;
                cA = a2; cB = b2;
            }
            if (wid == 0) STAMP(4 + 2 * it);
            {
                int tz = tid; asm volatile("" : "+v"(tz));
                const int ln = tz & 63, fr = ln & 15, fq = ln >> 4;
                const bool bad = (unsigned)__builtin_amdgcn_readfirstlane(__hip_atomic_load(tmo, RLX_AGENT)) == call;
                const bool own = ci < NO;
                const __amdgpu_buffer_rsrc_t ry = mkrsrc(myws + OFF_Y + (size_t)par * Y_BYTES, own ? (unsigned)(NO * 16 * 131072) : 0u);
                const __amdgpu_buffer_rsrc_t ro = mkrsrc(p.out, (unsigned)((size_t)M_ * N_ * 2));
                const unsigned yo = (unsigned)((ci * 16 + rb) * 131072 + wid * 16384 + ln * 16);
                const unsigned oo = (unsigned)((rb * 256 + wc * 32 + fq * 4) * N_ + cb * 256 + wr * 128 + fr * 8) * 2u;
#pragma unroll
                for (int bj = 0; bj < 2; ++bj)
#pragma unroll
                    for (int nf = 0; nf < 2; ++nf)
#pragma unroll
                        for (int j = 0; j < 4; ++j) {
                            u32x4 o;
#pragma unroll
                            for (int d = 0; d < 4; ++d)
                                o[d] = pack_bf16(silu(acc[d >> 1][bj][(d & 1) * 2][nf][j]), silu(acc[d >> 1][bj][(d & 1) * 2 + 1][nf][j]));
                            if (bad) o = (u32x4){0x7fc07fc0u, 0x7fc07fc0u, 0x7fc07fc0u, 0x7fc07fc0u};
                            __builtin_amdgcn_raw_buffer_store_b128(o, ro, oo + (unsigned)((bj * 128 + nf * 16 + j) * N_ * 2), 0, 16);
                            __builtin_amdgcn_raw_buffer_store_b128(o, ry, yo + ((bj * 2 + nf) * 4 + j) * 1024, 0, 17);
                        }
                if (own) {
                    WAIT_V(0);
                    if (ln < 7) {
                        const unsigned peer = (unsigned)(rank + 1 + ln) & 7u;
                        __hip_atomic_fetch_add(wptr(p.ws[peer], OFF_EXP + (size_t)(rank * 16 + 4 + ci) * 128), 1u, RLX_SYS);
                    }
                }
            }
            if (wid == 0) STAMP(5 + 2 * it);
#pragma unroll
            for (int a = 0; a < 2; ++a)
#pragma unroll
                for (int c = 0; c < 2; ++c)
#pragma unroll
                    for (int m = 0; m < 4; ++m)
#pragma unroll
                        for (int n = 0; n < 2; ++n) acc[a][c][m][n] = (f32x4){0.f, 0.f, 0.f, 0.f};
            ci = nci; cb = ncb;
        }
        WAIT_V(0);
        if (wr == 0) BAR;
    }
}

}

extern "C" void kernel_launch(int rank, int nranks, ncclComm_t comm,
                              hipStream_t stream,
                              void* const* d_in, const int* in_sizes, int n_in,
                              void* d_out, int out_size,
                              const int* in_dtypes, int out_dtype,
                              void* const* d_out_all,
                              void* const* d_ws_all, size_t ws_bytes) {
    (void)comm; (void)in_sizes; (void)in_dtypes; (void)out_dtype; (void)d_out_all; (void)out_size;
    if (nranks != NRANK || n_in < 2 || ws_bytes < WS_NEED || (unsigned)rank >= (unsigned)NRANK) {
        fprintf(stderr, "unexpected configuration\n");
        return;
    }
    HIP_CHECK(hipFuncSetAttribute((const void*)fused_gemm_allgather_silu, hipFuncAttributeMaxDynamicSharedMemorySize, LDS_BYTES));
    Params p;
    p.x = (const unsigned short*)d_in[0];
    p.w = (const unsigned short*)d_in[1];
    p.out = (unsigned short*)d_out;
    for (int i = 0; i < NRANK; ++i) p.ws[i] = (char*)d_ws_all[i];
    p.rank = rank;
    hipLaunchKernelGGL(fused_gemm_allgather_silu, dim3(NWG), dim3(512), LDS_BYTES, stream, p);
    HIP_CHECK(hipGetLastError());
}
```

```cpp
#include <hip/hip_runtime.h>
#include <rccl/rccl.h>
#include <cstdio>

#define HIP_CHECK(expr) do { hipError_t _e = (expr); if (_e != hipSuccess) { \
    fprintf(stderr, "HIP error %s at %s:%d\n", hipGetErrorString(_e), __FILE__, __LINE__); \
    return; } } while (0)

namespace {

#define LAS __attribute__((address_space(3)))
typedef short bf16x8 __attribute__((ext_vector_type(8)));
typedef float f32x4 __attribute__((ext_vector_type(4)));
typedef unsigned u32x4 __attribute__((ext_vector_type(4)));
typedef unsigned u32x2 __attribute__((ext_vector_type(2)));

constexpr int NRANK = 8;
constexpr int M_ = 4096, N_ = 16384, KSH = 512;
#define CFG_NO 4
#define CFG_CPX 4
#define CFG_GU 24
#define CFG_U 8
#define CFG_D 4
#define CFG_LDF "sc0 sc1"
#define CFG_SLEEP 16
constexpr int NO = CFG_NO;
constexpr int NREP = 64 - 8 * NO;
constexpr int NCB = NO + NREP;
constexpr int NWG = 256;
constexpr int CPX = CFG_CPX;
constexpr int NCOMMSLOT = 8 * CPX;
constexpr int NG = NWG - NCOMMSLOT;
constexpr int GU = CFG_GU;
constexpr int NW = CPX * 8;
constexpr int U = CFG_U;
constexpr int D = CFG_D;
constexpr int UPB = 64 / U;
constexpr int UPT = 128 / U;
static_assert(CPX == 3 || CPX == 4, "comm placement");
static_assert(GU <= NG / 8, "workers");
static_assert(D < U && D <= 4 && (U == 4 || U == 8), "pipeline shape");
constexpr int cb_end(int r) { return (2 * GU * (r + 1) + 3) / 4 < NCB ? (2 * GU * (r + 1) + 3) / 4 : NCB; }
constexpr int E0 = cb_end(0), E1 = cb_end(1), E2 = cb_end(2), E3 = cb_end(3);
static_assert(E3 == NCB && E0 >= NO, "at most four rounds, own blocks in the first");
constexpr int NCB1 = 8 * NO + (E0 - NO);
constexpr int NTASK = 320 - 16;
constexpr unsigned CNT_S0 = 64, CNT_S1 = 2 * NCB1 + 16, CNT_S2 = 2 * NCB1 + 32, CNT_S3 = (64 - NCB1) * 4, CNT_Y = 128;
static_assert(NTASK <= 2 * NG - 64, "tasks");
constexpr int NBLK_IN = 64 + 4 * NCB;
constexpr int NBLK1 = 4 * (16 + E0);
constexpr int NU_IN = NBLK_IN * UPB, NU = NU_IN + NO * 16 * UPT;
constexpr unsigned TPC = NU + 2 * NW;

constexpr size_t MiB = 1ull << 20;
constexpr size_t OFF_SEQ = 0;
constexpr size_t OFF_TMO = 32768;
constexpr size_t OFF_TK = 33792;
constexpr size_t OFF_EXP = 98304;
constexpr size_t OFF_LOCX = 45056;
constexpr size_t OFF_LOCW = 49152;
constexpr size_t OFF_IMPX = 57344;
constexpr size_t OFF_IMPW = 65536;
constexpr size_t CTL_BYTES = 131072;
constexpr size_t OFF_DBG = 262144;
constexpr size_t OFF_BUF = 1 * MiB;
constexpr size_t BUF_BYTES = 160 * MiB;
constexpr size_t W_OFF = 32 * MiB;
constexpr size_t OFF_Y = OFF_BUF + 2 * BUF_BYTES;
constexpr size_t Y_BYTES = 16 * MiB;
constexpr size_t WS_NEED = OFF_Y + 2 * Y_BYTES;
constexpr unsigned long long DEADLINE = 100000000ull;

constexpr int LDS_BYTES = 131072 + 64;

struct Params {
    const unsigned short* x;
    const unsigned short* w;
    unsigned short* out;
    char* ws[NRANK];
    int rank;
};

#define RLX_AGENT __ATOMIC_RELAXED, __HIP_MEMORY_SCOPE_AGENT
#define RLX_SYS __ATOMIC_RELAXED, __HIP_MEMORY_SCOPE_SYSTEM
#define WAIT_V(n) asm volatile("s_waitcnt vmcnt(" #n ")" ::: "memory")
#define WAIT_L(n) asm volatile("s_waitcnt lgkmcnt(" #n ")" ::: "memory")
#define BAR __builtin_amdgcn_s_barrier()
#define SCHED __builtin_amdgcn_sched_barrier(0)

__device__ __forceinline__ unsigned* wptr(char* base, size_t off) { return (unsigned*)(base + off); }
__device__ __forceinline__ __amdgpu_buffer_rsrc_t mkrsrc(const void* p, unsigned bytes) {
    return __builtin_amdgcn_make_buffer_rsrc((void*)p, 0, bytes, 0x00020000);
}
typedef int i32x4 __attribute__((ext_vector_type(4)));
__device__ __forceinline__ i32x4 mkdesc(const void* p, unsigned bytes) {
    const unsigned long long a = (unsigned long long)p;
    i32x4 d;
    d.x = __builtin_amdgcn_readfirstlane((int)(unsigned)a);
    d.y = __builtin_amdgcn_readfirstlane((int)((unsigned)(a >> 32) & 0xffffu));
    d.z = __builtin_amdgcn_readfirstlane((int)bytes);
    d.w = 0x00020000;
    return d;
}
template <int OFF> __device__ __forceinline__ void ld_sys(u32x4& v, const i32x4& d, unsigned voff) {
    asm volatile("buffer_load_dwordx4 %0, %1, %2, 0 offen offset:%3 " CFG_LDF : "=v"(v) : "v"(voff), "s"(d), "i"(OFF) : "memory");
}
__device__ __forceinline__ void st_wt(const u32x4& v, const i32x4& d, unsigned voff) {
    asm volatile("buffer_store_dwordx4 %0, %1, %2, 0 offen sc1\n\ts_nop 1" :: "v"(v), "v"(voff), "s"(d) : "memory");
}
__device__ __forceinline__ void add_noret(const i32x4& d, unsigned voff, unsigned one) {
    asm volatile("buffer_atomic_add %0, %1, %2, 0 offen" :: "v"(one), "v"(voff), "s"(d) : "memory");
}
__device__ __forceinline__ void add_ret(unsigned& r, const i32x4& d, unsigned voff) {
    asm volatile("buffer_atomic_add %0, %1, %2, 0 offen sc0" : "+v"(r) : "v"(voff), "s"(d) : "memory");
}
template <int N> __device__ __forceinline__ void wait_for(u32x4& v) { asm volatile("s_waitcnt vmcnt(%1)" : "+v"(v) : "i"(N) : "memory"); }
constexpr int younger_ops(int e) {
    int n = 0;
    const int s0 = e - D;
    for (int s = s0; s <= e; ++s) {
        const int m = (s + U) % U;
        if (s > s0) n += (m == U - D ? 1 : 0) + 1;
        if (s < e) n += 1 + (m == D ? 1 : 0);
    }
    return n;
}
template <int E> struct IC { static constexpr int value = E; };
__device__ __forceinline__ int gcb(int rank, int i) { return i < NO ? rank * NO + i : 8 * NO + (i - NO); }

__device__ __forceinline__ bool wave_wait(unsigned* f, unsigned need, unsigned* tmo, unsigned call) {
    unsigned v = __builtin_amdgcn_readfirstlane(__hip_atomic_load(f, RLX_SYS));
    if ((int)(v - need) >= 0) return true;
    const unsigned long long t0 = __builtin_amdgcn_s_memrealtime();
    for (;;) {
        __builtin_amdgcn_s_sleep(2);
        v = __builtin_amdgcn_readfirstlane(__hip_atomic_load(f, RLX_SYS));
        if ((int)(v - need) >= 0) return true;
        if ((unsigned)__builtin_amdgcn_readfirstlane(__hip_atomic_load(tmo, RLX_AGENT)) == call ||
            __builtin_amdgcn_s_memrealtime() - t0 > DEADLINE) {
            __hip_atomic_store(tmo, call, RLX_AGENT);
            return false;
        }
    }
}

__device__ __forceinline__ unsigned pack_bf16(float lo, float hi) {
    unsigned r; asm("v_cvt_pk_bf16_f32 %0, %1, %2" : "=v"(r) : "v"(lo), "v"(hi)); return r;
}
__device__ __forceinline__ float silu(float y) {
    const float c = __builtin_fminf(__builtin_fmaxf(y, -60.0f), 60.0f);
    const float e = __builtin_amdgcn_exp2f(c * -1.44269504088896f);
    return y * __builtin_amdgcn_rcpf(1.0f + e);
}

#define STAMP(slot) do {} while (0)
#define STAMPEND() do {} while (0)
#define STAMPC(slot) do {} while (0)

__global__ void __launch_bounds__(512, 2) fused_gemm_allgather_silu(const Params p) {
    extern __shared__ __attribute__((aligned(16))) unsigned char shm_[];
    LAS unsigned char* lds = (LAS unsigned char*)shm_;
    LAS unsigned* ctl = (LAS unsigned*)(lds + 131072);
    const int tid = threadIdx.x, lane = tid & 63;
    const int wid = __builtin_amdgcn_readfirstlane(tid >> 6);
    const int rank = p.rank;
    const int b = blockIdx.x;
    char* const myws = p.ws[rank];
    unsigned* const tmo = wptr(myws, OFF_TMO);
    if (wid == 0) STAMP(0);

    if (b < NCOMMSLOT) {
        const int link = CPX == 4 ? 2 * (b >> 3) + (b & 1) : ((b & 7) + 3 * (b >> 3)) & 7;
        if (link >= 7) return;
        const int src = (rank + 1 + link) & 7;
        const i32x4 dctl = mkdesc(myws, (unsigned)CTL_BYTES);
        const unsigned only0 = lane == 0 ? 0u : 0x80000000u;
        const unsigned tkoff = only0 | (unsigned)(OFF_TK + link * 128);
        unsigned t0v = 1u, tn = 1u;
        add_ret(t0v, dctl, tkoff);
        add_ret(tn, dctl, tkoff);
        const unsigned f0 = __hip_atomic_load(wptr(myws, OFF_EXP + (size_t)(src * 16) * 128), RLX_SYS);
        asm volatile("s_waitcnt vmcnt(0)" : "+v"(t0v), "+v"(tn) :: "memory");
        const unsigned tk0 = (unsigned)__builtin_amdgcn_readfirstlane(t0v);
        const unsigned call = tk0 / TPC + 1u, par = call & 1u;
        const unsigned tbase = (call - 1u) * TPC;
        const unsigned xoff = (unsigned)(OFF_BUF + (size_t)par * BUF_BYTES + (size_t)src * 4 * MiB);
        const unsigned woff = (unsigned)(OFF_BUF + (size_t)par * BUF_BYTES + W_OFF + (size_t)src * 16 * MiB);
        const unsigned yoff = (unsigned)(OFF_Y + (size_t)par * Y_BYTES);
        const i32x4 dsrc = mkdesc(p.ws[src], (unsigned)WS_NEED);
        const unsigned lane_in = (unsigned)lane * 16u, lane_out = (unsigned)((lane >> 4) * 4 * N_ + (lane & 15) * 8) * 2u;
        const unsigned one = 1u;
        int known = (int)((unsigned)__builtin_amdgcn_readfirstlane(f0) - call * CNT_S0) >= 0 ? 0 : -1;
        bool ok = true;
        auto resolve = [&](unsigned u, unsigned& so, unsigned& dof, unsigned& sig) {
            int stage;
            if (u < (unsigned)NU_IN) {
                const int blk = u / UPB, sub = u % UPB;
                int isx, j, idx;
                if (blk < NBLK1) {
                    j = blk / (16 + E0); const int r = blk % (16 + E0); isx = r < 16;
                    idx = isx ? ((r + 2 * link) & 15) : gcb(rank, (r - 16 + 2 * link) % E0);
                    stage = (isx && j == 0) ? 0 : (j < 2 ? 1 : 2);
                } else {
                    const int r2 = blk - 64;
                    const int lo = r2 < 4 * E1 ? E0 : (r2 < 4 * E2 ? E1 : E2), hi = r2 < 4 * E1 ? E1 : (r2 < 4 * E2 ? E2 : E3);
                    const int n = hi - lo, rr = r2 - 4 * lo;
                    j = rr / n; isx = 0; idx = gcb(rank, lo + (rr % n + 2 * link) % n);
                    stage = 3;
                }
                so = (isx ? xoff + (unsigned)(j * 16 + idx) * 65536u : woff + (unsigned)(idx * 4 + j) * 65536u) + (unsigned)sub * (U * 1024);
                dof = so;
                sig = isx ? (unsigned)OFF_IMPX + (unsigned)(j * 16 + idx) * 128u : (unsigned)OFF_IMPW + (unsigned)(idx * 4 + j) * 128u;
            } else if (u < (unsigned)NU) {
                const int uy = (int)u - NU_IN, t = uy / UPT, kb = (uy % UPT) * U;
                const int ci = t >> 4, rb = (t + 2 * link) & 15, w = kb >> 4, ix = kb & 15;
                const int m = rb * 256 + (ix >> 3) * 128 + (w & 3) * 32 + ((ix >> 2) & 1) * 16;
                const int n = (src * NO + ci) * 256 + (w >> 2) * 128;
                so = yoff + (unsigned)((ci * 16 + rb) * 131072 + kb * 1024);
                dof = (unsigned)(m * N_ + n) * 2u;
                sig = 0x80000000u;
                stage = 4 + ci;
            } else { so = 0x80000000u; dof = 0x80000000u; sig = 0x80000000u; stage = -1; }
            while (known < stage) {
                ++known;
                const unsigned cnt = known == 0 ? CNT_S0 : known == 1 ? CNT_S1 : known == 2 ? CNT_S2 : known == 3 ? CNT_S3 : CNT_Y;
                ok = wave_wait(wptr(myws, OFF_EXP + (size_t)(src * 16 + known) * 128), call * cnt, tmo, call) && ok;
                if (known >= 4) ok = ok && (unsigned)__builtin_amdgcn_readfirstlane(__hip_atomic_load(tmo, RLX_AGENT)) != call;
            }
        };
        unsigned u = tk0 - tbase;
        unsigned so, dof, sig, psig = 0x80000000u;
        resolve(u, so, dof, sig);
        if (wid == 0) STAMP(16);
        u32x4 v[U];
        {
            const unsigned sv = so + lane_in;
            auto pro = [&](auto ec) {
                constexpr int e = decltype(ec)::value;
                if constexpr (e < D) {
                    ld_sys<e * 1024>(v[e], dsrc, sv);
                    add_noret(dctl, 0x80000000u, one);
                    if constexpr (U - D + e == D) add_noret(dctl, 0x80000000u, one);
                }
            };
            pro(IC<0>{}); pro(IC<1>{}); pro(IC<2>{}); pro(IC<3>{});
        }
        while (u < (unsigned)NU) {
            const bool isout = u >= (unsigned)NU_IN;
            const i32x4 ddst = mkdesc(isout ? (const void*)p.out : (const void*)myws, isout ? (unsigned)((size_t)M_ * N_ * 2) : (unsigned)WS_NEED);
            const unsigned dv = dof + (isout ? lane_out : lane_in);
            const unsigned s0 = isout ? (unsigned)N_ * 2u : 1024u, s1 = isout ? (unsigned)N_ * 32u : 4096u;
            const unsigned sv = so + lane_in, sv4 = sv + 4096u;
            const bool okc = ok;
            unsigned un = 0, nso = 0, ndof = 0, nsig = 0, nsv = 0;
            auto step = [&](auto ec) {
                constexpr int e = decltype(ec)::value;
                if constexpr (e < U) {
                    if constexpr (e + D < U) {
                        ld_sys<((e + D) & 3) * 1024>(v[e + D], dsrc, e + D < 4 ? sv : sv4);
                    } else {
                        if constexpr (e + D == U) {
                            asm volatile("" : "+v"(tn));
                            un = (unsigned)__builtin_amdgcn_readfirstlane(tn) - tbase;
                            tn = 1u;
                            add_ret(tn, dctl, tkoff);
                            resolve(un, nso, ndof, nsig);
                            nsv = nso + lane_in;
                        }
                        ld_sys<(e + D - U) * 1024>(v[e + D - U], dsrc, nsv);
                    }
                    wait_for<younger_ops(e)>(v[e]);
                    if (!okc) v[e] = (u32x4){0x7fc07fc0u, 0x7fc07fc0u, 0x7fc07fc0u, 0x7fc07fc0u};
                    st_wt(v[e], ddst, dv + (e & 3) * s0 + (e >> 2) * s1);
                    if constexpr (e == D) add_noret(dctl, only0 | psig, one);
                }
            };
            step(IC<0>{}); step(IC<1>{}); step(IC<2>{}); step(IC<3>{});
            step(IC<4>{}); step(IC<5>{}); step(IC<6>{}); step(IC<7>{});
            psig = sig;
            u = un; so = nso; dof = ndof; sig = nsig;
        }
        WAIT_V(0);
        add_noret(dctl, only0 | psig, one);
        if (wid == 0) STAMP(3);
        STAMPEND();
        return;
    }

    const int g = b - NCOMMSLOT;
    unsigned call;
    {
        char* const peerws = p.ws[(unsigned)(rank + 1 + lane) & 7u];
        auto publish = [&](int stage, bool isx, int blk, unsigned cnt) {
            if (tid < 8) {
                const size_t o = tid < 7 ? OFF_EXP + (size_t)(rank * 16 + stage) * 128 : (isx ? OFF_LOCX : OFF_LOCW) + (size_t)blk * 128;
                __hip_atomic_fetch_add(wptr(peerws, o), tid < 7 ? 1u : cnt, RLX_SYS);
            }
        };
        u32x4 q0[2];
        if (g < 64) {
#pragma unroll
            for (int e = 0; e < 2; ++e) {
                const int pi = (g & 3) * 16 + wid * 2 + e;
                const int row = (g >> 2) * 256 + ((pi >> 1) & 15) * 16 + (lane & 15), kc = (pi >> 5) * 64 + (pi & 1) * 32 + (lane >> 4) * 8;
                q0[e] = *(const u32x4*)(p.x + (size_t)row * KSH + kc);
            }
        }
        if (tid == 0) ctl[0] = __hip_atomic_fetch_add(wptr(myws, OFF_SEQ + (size_t)b * 128), 1u, RLX_AGENT) + 1u;
        __syncthreads();
        call = __builtin_amdgcn_readfirstlane(ctl[0]);
        char* const bufp = myws + OFF_BUF + (size_t)(call & 1) * BUF_BYTES;
        const __amdgpu_buffer_rsrc_t rx = mkrsrc(bufp + (size_t)rank * 4 * MiB, 4u << 20);
        if (g < 64) {
#pragma unroll
            for (int e = 0; e < 2; ++e)
                __builtin_amdgcn_raw_buffer_store_b128(q0[e], rx, (unsigned)((g >> 2) * 65536 + ((g & 3) * 16 + wid * 2 + e) * 1024 + lane * 16), 0, 17);
            WAIT_V(0);
            __syncthreads();
            publish(0, true, g >> 2, 1u);
            if (wid == 0) STAMP(17);
        }
        for (int t = g; t < NTASK; t += (g >= 64 ? NG - 64 : NTASK)) {
            int isx = 0, blk, j, stage;
            if (t < NCB1) { blk = t; j = 0; stage = 1; }
            else {
                const int t1 = t - NCB1;
                if (t1 < 3 * (16 + NCB1)) { j = 1 + t1 / (16 + NCB1); const int r = t1 % (16 + NCB1); isx = r < 16; blk = isx ? r : r - 16; stage = j == 1 ? 1 : 2; }
                else { const int r = t1 - 3 * (16 + NCB1); blk = NCB1 + (r >> 2); j = r & 3; stage = 3; }
            }
            u32x4 v[8];
            if (isx) {
#pragma unroll
                for (int e = 0; e < 8; ++e) {
                    const int pi = wid * 8 + e;
                    const int row = blk * 256 + ((pi >> 1) & 15) * 16 + (lane & 15), kc = j * 128 + (pi >> 5) * 64 + (pi & 1) * 32 + (lane >> 4) * 8;
                    v[e] = *(const u32x4*)(p.x + (size_t)row * KSH + kc);
                }
#pragma unroll
                for (int e = 0; e < 8; ++e)
                    __builtin_amdgcn_raw_buffer_store_b128(v[e], rx, (unsigned)((j * 16 + blk) * 65536 + (wid * 8 + e) * 1024 + lane * 16), 0, 17);
            } else {
                const __amdgpu_buffer_rsrc_t rw = mkrsrc(bufp + W_OFF + (size_t)rank * 16 * MiB, 16u << 20);
                const int o = wid * 2 + (lane >> 5);
                const unsigned short* src = p.w + (size_t)(j * 128 + o * 8) * N_ + blk * 256 + (lane & 31) * 8;
#pragma unroll
                for (int e = 0; e < 8; ++e) v[e] = *(const u32x4*)(src + (size_t)e * N_);
                const unsigned lo = (unsigned)((blk * 4 + j) * 65536 + (o >> 3) * 32768 + ((lane >> 4) & 1) * 8192 + ((o >> 2) & 1) * 1024 + ((o & 3) * 16 + (lane & 15)) * 16);
#pragma unroll
                for (int f = 0; f < 8; ++f) {
                    u32x4 ov;
#pragma unroll
                    for (int q = 0; q < 4; ++q)
                        ov[q] = __builtin_amdgcn_perm(v[2 * q + 1][f >> 1], v[2 * q][f >> 1], (f & 1) ? 0x07060302u : 0x05040100u);
                    __builtin_amdgcn_raw_buffer_store_b128(ov, rw, lo + (f >> 2) * 16384 + (f & 3) * 2048, 0, 17);
                }
            }
            WAIT_V(0);
            __syncthreads();
            publish(stage, isx, blk, 4u);
        }
    }
    if (wid == 0) STAMP(1);
    const unsigned par = call & 1u;
    {
        if ((g >> 3) >= GU) return;
        const int li0 = GU * ((g >> 2) & 1) + (g >> 3), rq = 4 * (g & 3);
        const int wr = wid >> 2, wc = wid & 3;
        const char* const bufx = myws + OFF_BUF + (size_t)par * BUF_BYTES;
        const char* const bufw = bufx + W_OFF;
        const unsigned voff = (unsigned)tid * 16u;
        const unsigned ldsw = (unsigned)wid * 1024u;
        const int aoff = wr * 8192 + lane * 16, boff = wc * 4096 + lane * 16;
        const int ntile = (4 * NCB - li0 + 2 * GU - 1) / (2 * GU);
#define SA(bb, h) (((bb) * 2 + (h)) * 16384)
#define SB(bb, h) ((4 + (bb) * 2 + (h)) * 16384)
#define STAGE(bufoff, gbase) do { _Pragma("unroll") for (int _i = 0; _i < 2; ++_i) \
        __builtin_amdgcn_global_load_lds((const unsigned*)((gbase) + voff + _i * 8192), (LAS unsigned*)(lds + (bufoff) + ldsw + _i * 8192), 16, 0, 0); } while (0)
#define LDA(dst, bb, h) do { _Pragma("unroll") for (int m = 0; m < 4; ++m) _Pragma("unroll") for (int k = 0; k < 2; ++k) dst[m][k] = *(const LAS bf16x8*)(lds + SA(bb, h) + aoff + m * 2048 + k * 1024); } while (0)
#define LDB(dst, bb, h) do { _Pragma("unroll") for (int n = 0; n < 2; ++n) _Pragma("unroll") for (int k = 0; k < 2; ++k) dst[n][k] = *(const LAS bf16x8*)(lds + SB(bb, h) + boff + n * 2048 + k * 1024); } while (0)
#define MMA(ai, bj, At, Bt) do { __builtin_amdgcn_s_setprio(1); _Pragma("unroll") for (int m = 0; m < 4; ++m) _Pragma("unroll") for (int n = 0; n < 2; ++n) _Pragma("unroll") for (int k = 0; k < 2; ++k) \
        acc[ai][bj][m][n] = __builtin_amdgcn_mfma_f32_16x16x32_bf16(Bt[n][k], At[m][k], acc[ai][bj][m][n], 0, 0, 0); __builtin_amdgcn_s_setprio(0); } while (0)

        auto pair_src = [&](int pp) { return pp < 4 ? rank : ((rank + 1 + (pp - 4) % 7) & 7); };
        auto pair_j = [&](int pp) { return pp < 4 ? pp : (pp - 4) / 7; };
        auto wbase = [&](int cb, int pp) { return bufw + (size_t)pair_src(pp) * 16 * MiB + (size_t)(cb * 4 + pair_j(pp)) * 65536; };
        auto xbase = [&](int rb, int pp) { return bufx + (size_t)pair_src(pp) * 4 * MiB + (size_t)(pair_j(pp) * 16 + rb) * 65536; };
        auto ready = [&](int rb, int cb, int seg) -> int {
            if (wid == 0) {
                const int l = lane & 7, side = (lane >> 3) & 1;
                const bool act = lane < 16 && l < 5;
                size_t o;
                if (side == 0) o = l == 0 ? OFF_LOCX + (size_t)rb * 128 : OFF_IMPX + (size_t)(((l - 1) & 3) * 16 + rb) * 128;
                else o = l == 0 ? OFF_LOCW + (size_t)cb * 128 : OFF_IMPW + (size_t)(cb * 4 + ((l - 1) & 3)) * 128;
                const unsigned need = call * (l == 0 ? 16u : (unsigned)(7 * UPB));
                unsigned* f = wptr(myws, o);
                const unsigned long long t0 = __builtin_amdgcn_s_memrealtime();
                int nrdy;
                for (;;) {
                    const unsigned v = act ? __hip_atomic_load(f, RLX_AGENT) : need;
                    const unsigned long long bal = __ballot((int)(v - need) >= 0);
                    const unsigned both = (unsigned)(bal & (bal >> 8)) & 0x1fu;
                    nrdy = __builtin_ctz(~(both | ((1u << seg) - 1u)));
                    if (nrdy > seg) break;
                    if ((unsigned)__builtin_amdgcn_readfirstlane(__hip_atomic_load(tmo, RLX_AGENT)) == call ||
                        __builtin_amdgcn_s_memrealtime() - t0 > DEADLINE) {
                        __hip_atomic_store(tmo, call, RLX_AGENT);
                        nrdy = 5; break;
                    }
                    __builtin_amdgcn_s_sleep(CFG_SLEEP);
                }
                __builtin_amdgcn_fence(__ATOMIC_ACQUIRE, "agent");
                if (lane == 0) ctl[1] = (unsigned)nrdy;
                asm volatile("s_waitcnt vmcnt(0) lgkmcnt(0)" ::: "memory");
            }
            asm volatile("" ::: "memory"); BAR; asm volatile("" ::: "memory");
            const int nr = __builtin_amdgcn_readfirstlane(*(volatile LAS unsigned*)&ctl[1]);
            WAIT_L(0);
            return nr >= 5 ? 32 : 4 + 7 * (nr - 1);
        };

        f32x4 acc[2][2][4][2];
#pragma unroll
        for (int a = 0; a < 2; ++a)
#pragma unroll
            for (int c = 0; c < 2; ++c)
#pragma unroll
                for (int m = 0; m < 4; ++m)
#pragma unroll
                    for (int n = 0; n < 2; ++n) acc[a][c][m][n] = (f32x4){0.f, 0.f, 0.f, 0.f};
        bf16x8 At[4][2], B0[2][2], B1[2][2];
        int li = li0, ci = li >> 2, rb = rq + (li & 3), cb = gcb(rank, ci);
        int lim = ready(rb, cb, 0);
        const char* cA = wbase(cb, 0); const char* cB = xbase(rb, 0);
        STAGE(SB(0, 0), cB); STAGE(SA(0, 0), cA); STAGE(SB(0, 1), cB + 16384); STAGE(SA(0, 1), cA + 16384);
        if (wr == 1) BAR;
        WAIT_V(4); BAR;
        STAGE(SB(1, 0), cB + 32768); STAGE(SA(1, 0), cA + 32768); STAGE(SB(1, 1), cB + 49152);
        WAIT_V(6); BAR;
        if (wid == 0) STAMP(2);
        for (int it = 0; it < ntile; ++it) {
            const bool has_next = it + 1 < ntile;
            const int nli = has_next ? li + 2 * GU : li, nci = nli >> 2, nrb = rq + (nli & 3), ncb = gcb(rank, nci);
            for (int pp = 0; pp < 32; ++pp) {
                const bool last = pp == 31;
                const char* a2; const char* b2;
                if (!last) {
                    if (pp + 1 >= lim) lim = ready(rb, cb, 1 + (pp + 1 - 4) / 7);
                    a2 = wbase(cb, pp + 1); b2 = xbase(rb, pp + 1);
                } else {
                    if (has_next) lim = ready(nrb, ncb, 0);
                    a2 = wbase(ncb, 0); b2 = xbase(nrb, 0);
                }
                LDB(B0, 0, 0); SCHED; LDA(At, 0, 0); STAGE(SA(1, 1), cA + 49152);
                WAIT_L(8); BAR; WAIT_L(0); MMA(0, 0, At, B0); BAR; SCHED;
                LDB(B1, 0, 1); STAGE(SB(0, 0), b2);
                BAR; WAIT_L(0); MMA(0, 1, At, B1); BAR;
                LDA(At, 0, 1); STAGE(SA(0, 0), a2);
                BAR; WAIT_L(0); MMA(1, 0, At, B0); BAR; SCHED;
                STAGE(SB(0, 1), b2 + 16384);
                WAIT_V(6); BAR; MMA(1, 1, At, B1); BAR;
                LDB(B0, 1, 0); SCHED; LDA(At, 1, 0); STAGE(SA(0, 1), a2 + 16384);
                WAIT_L(8); BAR; WAIT_L(0); MMA(0, 0, At, B0); BAR; SCHED;
                LDB(B1, 1, 1); STAGE(SB(1, 0), b2 + 32768);
                BAR; WAIT_L(0); MMA(0, 1, At, B1); BAR;
                LDA(At, 1, 1); STAGE(SA(1, 0), a2 + 32768);
                BAR; WAIT_L(0); MMA(1, 0, At, B0); BAR; SCHED;
                STAGE(SB(1, 1), b2 + 49152);
                WAIT_V(6); BAR; MMA(1, 1, At, B1); BAR;
                cA = a2; cB = b2;
            }
            if (wid == 0) { STAMP(4 + 2 * it); STAMPC(44 + 2 * it); }
            {
                int tz = tid; asm volatile("" : "+v"(tz));
                const int ln = tz & 63, fr = ln & 15, fq = ln >> 4;
                const bool bad = (unsigned)__builtin_amdgcn_readfirstlane(__hip_atomic_load(tmo, RLX_AGENT)) == call;
                const bool own = ci < NO;
                const __amdgpu_buffer_rsrc_t ry = mkrsrc(myws + OFF_Y + (size_t)par * Y_BYTES, own ? (unsigned)(NO * 16 * 131072) : 0u);
                const __amdgpu_buffer_rsrc_t ro = mkrsrc(p.out, (unsigned)((size_t)M_ * N_ * 2));
                const unsigned yo = (unsigned)((ci * 16 + rb) * 131072 + wid * 16384 + ln * 16);
                const unsigned oo = (unsigned)((rb * 256 + wc * 32 + fq * 4) * N_ + cb * 256 + wr * 128 + fr * 8) * 2u;
#pragma unroll
                for (int bj = 0; bj < 2; ++bj)
#pragma unroll
                    for (int nf = 0; nf < 2; ++nf)
#pragma unroll
                        for (int j = 0; j < 4; ++j) {
                            u32x4 o;
#pragma unroll
                            for (int d = 0; d < 4; ++d)
                                o[d] = pack_bf16(silu(acc[d >> 1][bj][(d & 1) * 2][nf][j]), silu(acc[d >> 1][bj][(d & 1) * 2 + 1][nf][j]));
                            if (bad) o = (u32x4){0x7fc07fc0u, 0x7fc07fc0u, 0x7fc07fc0u, 0x7fc07fc0u};
                            __builtin_amdgcn_raw_buffer_store_b128(o, ro, oo + (unsigned)((bj * 128 + nf * 16 + j) * N_ * 2), 0, 16);
                            __builtin_amdgcn_raw_buffer_store_b128(o, ry, yo + ((bj * 2 + nf) * 4 + j) * 1024, 0, 17);
                        }
                if (own) {
                    WAIT_V(0);
                    if (ln < 7) {
                        const unsigned peer = (unsigned)(rank + 1 + ln) & 7u;
                        __hip_atomic_fetch_add(wptr(p.ws[peer], OFF_EXP + (size_t)(rank * 16 + 4 + ci) * 128), 1u, RLX_SYS);
                    }
                }
            }
            if (wid == 0) { STAMP(5 + 2 * it); STAMPC(45 + 2 * it); }
#pragma unroll
            for (int a = 0; a < 2; ++a)
#pragma unroll
                for (int c = 0; c < 2; ++c)
#pragma unroll
                    for (int m = 0; m < 4; ++m)
#pragma unroll
                        for (int n = 0; n < 2; ++n) acc[a][c][m][n] = (f32x4){0.f, 0.f, 0.f, 0.f};
            li = nli; ci = nci; rb = nrb; cb = ncb;
        }
        WAIT_V(0);
        if (wr == 0) BAR;
    }
}

}

extern "C" void kernel_launch(int rank, int nranks, ncclComm_t comm,
                              hipStream_t stream,
                              void* const* d_in, const int* in_sizes, int n_in,
                              void* d_out, int out_size,
                              const int* in_dtypes, int out_dtype,
                              void* const* d_out_all,
                              void* const* d_ws_all, size_t ws_bytes) {
    (void)comm; (void)in_sizes; (void)in_dtypes; (void)out_dtype; (void)d_out_all; (void)out_size;
    if (nranks != NRANK || n_in < 2 || ws_bytes < WS_NEED || (unsigned)rank >= (unsigned)NRANK) {
        fprintf(stderr, "unexpected configuration\n");
        return;
    }
    HIP_CHECK(hipFuncSetAttribute((const void*)fused_gemm_allgather_silu, hipFuncAttributeMaxDynamicSharedMemorySize, LDS_BYTES));
    Params p;
    p.x = (const unsigned short*)d_in[0];
    p.w = (const unsigned short*)d_in[1];
    p.out = (unsigned short*)d_out;
    for (int i = 0; i < NRANK; ++i) p.ws[i] = (char*)d_ws_all[i];
    p.rank = rank;
    hipLaunchKernelGGL(fused_gemm_allgather_silu, dim3(NWG), dim3(512), LDS_BYTES, stream, p);
    HIP_CHECK(hipGetLastError());
}
```

```cpp
#include <hip/hip_runtime.h>
#include <rccl/rccl.h>
#include <cstdio>

#define HIP_CHECK(expr) do { hipError_t _e = (expr); if (_e != hipSuccess) { \
    fprintf(stderr, "HIP error %s at %s:%d\n", hipGetErrorString(_e), __FILE__, __LINE__); \
    return; } } while (0)

namespace {

#define LAS __attribute__((address_space(3)))
typedef short bf16x8 __attribute__((ext_vector_type(8)));
typedef float f32x4 __attribute__((ext_vector_type(4)));
typedef unsigned u32x4 __attribute__((ext_vector_type(4)));
typedef unsigned u32x2 __attribute__((ext_vector_type(2)));

constexpr int NRANK = 8;
constexpr int M_ = 4096, N_ = 16384, KSH = 512;
#define CFG_NO 3
#define CFG_CPX 4
#define CFG_GU 28
#define CFG_U 8
#define CFG_D 4
#define CFG_EC 2
#define CFG_FI 0
#define CFG_WPC 4
#define CFG_LAY 1
#define CFG_LDF "sc0 sc1"
#define CFG_SLEEP 16
constexpr int NO = CFG_NO;
constexpr int NREP = 64 - 8 * NO;
constexpr int NCB = NO + NREP;
constexpr int NWG = 256;
constexpr int CPX = CFG_CPX;
constexpr int NCOMMSLOT = 8 * CPX;
constexpr int NG = NWG - NCOMMSLOT;
constexpr int GU = CFG_GU;
constexpr int WPC = CFG_WPC;
constexpr int NW = CPX * WPC;
constexpr int U = CFG_U;
constexpr int D = CFG_D;
constexpr int UPI = 32 / U;
constexpr int UPT = 128 / U;
constexpr int TS = U - 2 * D;
constexpr int FI = CFG_FI;
constexpr int EC = CFG_EC;
static_assert(CPX == 3 || CPX == 4, "comm placement");
static_assert(GU <= NG / 8 && 4 * EC <= GU, "workers");
static_assert(2 * D <= U && D <= 4 && (U == 4 || U == 8), "pipeline shape");
constexpr int LLEN = 4 * (NCB - EC), RFULL = LLEN / (2 * GU), LREM = LLEN % (2 * GU), NREG = 2 * GU - 4 * EC;
constexpr int LEXC = LREM > NREG ? LREM - NREG : 0;
static_assert(2 * LEXC <= 4 * EC, "half tiles");
constexpr int rnd_end(int r) { return EC + (2 * GU * (r + 1) + 3) / 4 < NCB ? EC + (2 * GU * (r + 1) + 3) / 4 : NCB; }
constexpr int B1 = EC ? EC : rnd_end(0), B2 = rnd_end(EC ? 0 : 1), B3 = rnd_end(EC ? 1 : 2), B4 = rnd_end(EC ? 2 : 3);
static_assert(B4 == NCB && rnd_end(0) >= NO, "group count, own blocks in the first round");
constexpr int NCB1 = 8 * NO + (rnd_end(0) - NO);
constexpr int NTASK = 320 - 16;
constexpr unsigned CNT_S0 = 64, CNT_S1 = 2 * NCB1 + 16, CNT_S2 = 2 * NCB1 + 32, CNT_S3 = (64 - NCB1) * 4, CNT_Y = 128;
static_assert(NTASK <= 2 * NG - 64, "tasks");
constexpr int NIMG_IN = 128 + 8 * NCB;
constexpr int NU_IN = NIMG_IN * UPI, NU = NU_IN + NO * 16 * UPT;
constexpr unsigned TPC = NU + NW;

constexpr size_t MiB = 1ull << 20;
constexpr size_t OFF_SEQ = 0;
constexpr size_t OFF_TMO = 32768;
constexpr size_t OFF_TK = 33792;
constexpr size_t OFF_EXP = 49152;
constexpr size_t OFF_RDY = 65536;
constexpr size_t CTL_BYTES = 131072;
constexpr size_t OFF_DBG = 262144;
constexpr size_t OFF_BUF = 1 * MiB;
constexpr unsigned BSTR = 32768u, TSTR = 80u * 32768u;
constexpr size_t BUF_BYTES = 160 * MiB;
constexpr size_t OFF_Y = OFF_BUF + 2 * BUF_BYTES;
constexpr size_t Y_BYTES = 16 * MiB;
constexpr size_t WS_NEED = OFF_Y + 2 * Y_BYTES;
constexpr unsigned long long DEADLINE = 100000000ull;

constexpr int LDS_BYTES = 131072 + 768;
constexpr int NST = 4 + NO;

struct Params {
    const unsigned short* x;
    const unsigned short* w;
    unsigned short* out;
    char* ws[NRANK];
    int rank;
};

#define RLX_AGENT __ATOMIC_RELAXED, __HIP_MEMORY_SCOPE_AGENT
#define RLX_SYS __ATOMIC_RELAXED, __HIP_MEMORY_SCOPE_SYSTEM
#define WAIT_V(n) asm volatile("s_waitcnt vmcnt(" #n ")" ::: "memory")
#define WAIT_L(n) asm volatile("s_waitcnt lgkmcnt(" #n ")" ::: "memory")
#define BAR __builtin_amdgcn_s_barrier()
#define SCHED __builtin_amdgcn_sched_barrier(0)

__device__ __forceinline__ unsigned* wptr(char* base, size_t off) { return (unsigned*)(base + off); }
__device__ __forceinline__ __amdgpu_buffer_rsrc_t mkrsrc(const void* p, unsigned bytes) {
    return __builtin_amdgcn_make_buffer_rsrc((void*)p, 0, bytes, 0x00020000);
}
typedef int i32x4 __attribute__((ext_vector_type(4)));
__device__ __forceinline__ i32x4 mkdesc(const void* p, unsigned bytes) {
    const unsigned long long a = (unsigned long long)p;
    i32x4 d;
    d.x = __builtin_amdgcn_readfirstlane((int)(unsigned)a);
    d.y = __builtin_amdgcn_readfirstlane((int)((unsigned)(a >> 32) & 0xffffu));
    d.z = __builtin_amdgcn_readfirstlane((int)bytes);
    d.w = 0x00020000;
    return d;
}
template <int OFF> __device__ __forceinline__ void ld_sys(u32x4& v, const i32x4& d, unsigned voff) {
    asm volatile("buffer_load_dwordx4 %0, %1, %2, 0 offen offset:%3 " CFG_LDF : "=v"(v) : "v"(voff), "s"(d), "i"(OFF) : "memory");
}
__device__ __forceinline__ void st_wt(const u32x4& v, const i32x4& d, unsigned voff) {
    asm volatile("buffer_store_dwordx4 %0, %1, %2, 0 offen sc1\n\ts_nop 1" :: "v"(v), "v"(voff), "s"(d) : "memory");
}
__device__ __forceinline__ void add_noret(const i32x4& d, unsigned voff, unsigned one) {
    asm volatile("buffer_atomic_add %0, %1, %2, 0 offen" :: "v"(one), "v"(voff), "s"(d) : "memory");
}
__device__ __forceinline__ void add_ret(unsigned& r, const i32x4& d, unsigned voff) {
    asm volatile("buffer_atomic_add %0, %1, %2, 0 offen sc0" : "+v"(r) : "v"(voff), "s"(d) : "memory");
}
template <int N> __device__ __forceinline__ void wait_for(u32x4& v) { asm volatile("s_waitcnt vmcnt(%1)" : "+v"(v) : "i"(N) : "memory"); }
__device__ __forceinline__ void ld_word(unsigned& r, const i32x4& d, unsigned voff) {
    asm volatile("buffer_load_dword %0, %1, %2, 0 offen sc0 sc1" : "=v"(r) : "v"(voff), "s"(d) : "memory");
}
constexpr int younger_ops(int e) {
    int n = 0;
    const int s0 = e - D;
    for (int s = s0; s <= e; ++s) {
        const int m = (s + U) % U;
        if (s > s0) n += (m == TS ? 2 : 0) + 1;
        if (s < e) n += 1 + (m == D ? 1 : 0);
    }
    return n;
}
constexpr int younger_tk() {
    int n = 0;
    for (int s = TS; s < U - D; ++s) n += 2 + (s == D ? 1 : 0);
    return n;
}
template <int E> struct IC { static constexpr int value = E; };
__device__ __forceinline__ int gcb(int rank, int i) { return i < NO ? rank * NO + i : 8 * NO + (i - NO); }

__device__ __forceinline__ unsigned pack_bf16(float lo, float hi) {
    unsigned r; asm("v_cvt_pk_bf16_f32 %0, %1, %2" : "=v"(r) : "v"(lo), "v"(hi)); return r;
}
__device__ __forceinline__ float silu(float y) {
    const float c = __builtin_fminf(__builtin_fmaxf(y, -60.0f), 60.0f);
    const float e = __builtin_amdgcn_exp2f(c * -1.44269504088896f);
    return y * __builtin_amdgcn_rcpf(1.0f + e);
}

#define STAMP(slot) do {} while (0)
#define STAMPEND() do {} while (0)
#define STAMPC(slot) do {} while (0)

__global__ void __launch_bounds__(512, 2) fused_gemm_allgather_silu(const Params p) {
    extern __shared__ __attribute__((aligned(16))) unsigned char shm_[];
    LAS unsigned char* lds = (LAS unsigned char*)shm_;
    LAS unsigned* ctl = (LAS unsigned*)(lds + 131072);
    const int tid = threadIdx.x, lane = tid & 63;
    const int wid = __builtin_amdgcn_readfirstlane(tid >> 6);
    const int rank = p.rank;
    const int b = blockIdx.x;
    char* const myws = p.ws[rank];
    unsigned* const tmo = wptr(myws, OFF_TMO);
    if (wid == 0) STAMP(0);

    if (b < NCOMMSLOT) {
        const int link = CPX == 4 ? 2 * (b >> 3) + (b & 1) : ((b & 7) + 3 * (b >> 3)) & 7;
        if (link >= 7 || wid >= WPC) return;
        const int src = (rank + 1 + link) & 7;
        const i32x4 dctl = mkdesc(myws, (unsigned)CTL_BYTES);
        const unsigned only0 = lane == 0 ? 0u : 0x80000000u;
        const unsigned tkoff = only0 | (unsigned)(OFF_TK + link * 128);
        unsigned tn = 1u;
        add_ret(tn, dctl, tkoff);
        const unsigned floff = lane < NST ? (unsigned)(OFF_EXP + src * 128 + lane * 4) : (lane == 15 ? (unsigned)OFF_TMO : 0x80000000u);
        unsigned fv;
        ld_word(fv, dctl, floff);
        asm volatile("s_waitcnt vmcnt(0)" : "+v"(tn), "+v"(fv) :: "memory");
        const unsigned tk0 = (unsigned)__builtin_amdgcn_readfirstlane(tn);
        const unsigned call = tk0 / TPC + 1u, par = call & 1u;
        const unsigned tbase = (call - 1u) * TPC;
        const unsigned boffs = (unsigned)(OFF_BUF + (size_t)par * BUF_BYTES);
        const unsigned yoff = (unsigned)(OFF_Y + (size_t)par * Y_BYTES);
        const i32x4 dsrc = mkdesc(p.ws[src], (unsigned)WS_NEED);
        const unsigned lane_in = (unsigned)lane * 16u, lane_out = (unsigned)((lane >> 4) * 4 * N_ + (lane & 15) * 8) * 2u;
        const unsigned one = 1u;
        const unsigned fneed = call * (lane == 0 ? CNT_S0 : lane == 1 ? CNT_S1 : lane == 2 ? CNT_S2 : lane == 3 ? CNT_S3 : CNT_Y);
        int known = 0;
        bool ok = true;
        auto update = [&]() {
            known = __builtin_ctz(~(unsigned)__ballot(lane < NST && (int)(fv - fneed) >= 0));
            if ((unsigned)__builtin_amdgcn_readlane((int)fv, 15) == call) { ok = false; known = NST; }
        };
        update();
        auto resolve = [&](unsigned u, unsigned& so, unsigned& dof, unsigned& sig) {
            int stage;
            if (u < (unsigned)NU_IN) {
                const int img = u / UPI, sub = u % UPI;
                int kt, blk;
                int q = img - 8 * (16 + B1);
                blk = -1;
                if (img < 8 * (16 + B1 + FI)) {
                    kt = img / (16 + B1 + FI); const int r = img % (16 + B1 + FI);
                    if (r < 16 + B1) blk = r < 16 ? ((r + 2 * link) & 15) : 16 + gcb(rank, (r - 16 + 2 * link) % B1);
                    q = kt * FI + (r - 16 - B1);
                }
                if (blk < 0) {
                    const int r2 = q + 8 * B1;
                    const int lo = r2 < 8 * B2 ? B1 : (r2 < 8 * B3 ? B2 : B3), hi = r2 < 8 * B2 ? B2 : (r2 < 8 * B3 ? B3 : B4);
                    const int n = hi - lo, rr = r2 - 8 * lo;
                    kt = rr / n; blk = 16 + gcb(rank, lo + (rr % n + 2 * link) % n);
                }
                stage = blk >= 16 + NCB1 ? 3 : ((blk < 16 && kt < 2) ? 0 : (kt < 4 ? 1 : 2));
                const unsigned o = boffs + (unsigned)blk * BSTR + (unsigned)sub * (U * 1024);
                so = o + (unsigned)kt * TSTR;
                dof = o + (unsigned)(8 + 7 * kt + link) * TSTR;
                sig = (unsigned)OFF_RDY + (unsigned)(blk * 128 + 4 + kt * 4);
            } else if (u < (unsigned)NU) {
                const int uy = (int)u - NU_IN, t = uy / UPT, kb = (uy % UPT) * U;
                const int ci = t >> 4, rb = (t + 2 * link) & 15, w = kb >> 4, ix = kb & 15;
                const int m = rb * 256 + (ix >> 3) * 128 + (w & 3) * 32 + ((ix >> 2) & 1) * 16;
                const int n = (src * NO + ci) * 256 + (w >> 2) * 128;
                so = yoff + (unsigned)((ci * 16 + rb) * 131072 + kb * 1024);
                dof = (unsigned)(m * N_ + n) * 2u;
                sig = 0x80000000u;
                stage = 4 + ci;
            } else { so = 0x80000000u; dof = 0x80000000u; sig = 0x80000000u; stage = -1; }
            if (known <= stage) {
                const unsigned long long t0 = __builtin_amdgcn_s_memrealtime();
                for (;;) {
                    ld_word(fv, dctl, floff);
                    asm volatile("s_waitcnt vmcnt(0)" : "+v"(fv) :: "memory");
                    update();
                    if (known > stage) break;
                    if (__builtin_amdgcn_s_memrealtime() - t0 > DEADLINE) {
                        __hip_atomic_store(tmo, call, RLX_AGENT);
                        ok = false; known = NST;
                        break;
                    }
                    __builtin_amdgcn_s_sleep(2);
                }
            }
        };
        unsigned u = tk0 - tbase;
        unsigned so, dof, sig, psig = 0x80000000u;
        resolve(u, so, dof, sig);
        if (wid == 0) STAMP(16);
        u32x4 v[U];
        {
            const unsigned sv = so + lane_in;
            auto pro = [&](auto ec) {
                constexpr int e = decltype(ec)::value;
                if constexpr (e < D) {
                    ld_sys<e * 1024>(v[e], dsrc, sv);
                    add_noret(dctl, 0x80000000u, one);
                    if constexpr (U - D + e == D) add_noret(dctl, 0x80000000u, one);
                    static_assert(TS < U - D, "no ticket draw inside the last D steps");
                }
            };
            pro(IC<0>{}); pro(IC<1>{}); pro(IC<2>{}); pro(IC<3>{});
        }
        while (u < (unsigned)NU) {
            const bool isout = u >= (unsigned)NU_IN;
            const i32x4 ddst = mkdesc(isout ? (const void*)p.out : (const void*)myws, isout ? (unsigned)((size_t)M_ * N_ * 2) : (unsigned)WS_NEED);
            const unsigned dv = dof + (isout ? lane_out : lane_in);
            const unsigned s0 = isout ? (unsigned)N_ * 2u : 1024u, s1 = isout ? (unsigned)N_ * 32u : 4096u;
            const unsigned sv = so + lane_in, sv4 = sv + 4096u;
            const bool okc = ok;
            unsigned un = 0, nso = 0, ndof = 0, nsig = 0, nsv = 0;
            auto step = [&](auto ec) {
                constexpr int e = decltype(ec)::value;
                if constexpr (e < U) {
                    if constexpr (e == TS) { tn = 1u; add_ret(tn, dctl, tkoff); ld_word(fv, dctl, floff); }
                    if constexpr (e + D < U) {
                        ld_sys<((e + D) & 3) * 1024>(v[e + D], dsrc, e + D < 4 ? sv : sv4);
                    } else {
                        if constexpr (e + D == U) {
                            asm volatile("s_waitcnt vmcnt(%2)" : "+v"(tn), "+v"(fv) : "i"(younger_tk()) : "memory");
                            un = (unsigned)__builtin_amdgcn_readfirstlane(tn) - tbase;
                            if (ok) update();
                            resolve(un, nso, ndof, nsig);
                            nsv = nso + lane_in;
                        }
                        ld_sys<(e + D - U) * 1024>(v[e + D - U], dsrc, nsv);
                    }
                    wait_for<younger_ops(e)>(v[e]);
                    if (!okc) v[e] = (u32x4){0x7fc07fc0u, 0x7fc07fc0u, 0x7fc07fc0u, 0x7fc07fc0u};
                    st_wt(v[e], ddst, dv + (e & 3) * s0 + (e >> 2) * s1);
                    if constexpr (e == D) add_noret(dctl, only0 | psig, one);
                }
            };
            step(IC<0>{}); step(IC<1>{}); step(IC<2>{}); step(IC<3>{});
            step(IC<4>{}); step(IC<5>{}); step(IC<6>{}); step(IC<7>{});
            psig = sig;
            u = un; so = nso; dof = ndof; sig = nsig;
        }
        WAIT_V(0);
        add_noret(dctl, only0 | psig, one);
        if (wid == 0) STAMP(3);
        STAMPEND();
        return;
    }

    const int g = b - NCOMMSLOT;
    unsigned call;
    {
        char* const peerws = p.ws[(unsigned)(rank + 1 + lane) & 7u];
        auto publish = [&](int stage, bool isx, int blk, unsigned cnt) {
            if (tid < 8) {
                const size_t o = tid < 7 ? OFF_EXP + (size_t)(rank * 128 + stage * 4) : OFF_RDY + (size_t)(isx ? blk : 16 + blk) * 128;
                __hip_atomic_fetch_add(wptr(peerws, o), tid < 7 ? 1u : cnt, RLX_SYS);
            }
        };
        u32x4 q0[2];
        if (g < 64) {
#pragma unroll
            for (int e = 0; e < 2; ++e) {
                const int pi = (g & 3) * 16 + wid * 2 + e;
                const int row = (g >> 2) * 256 + ((pi >> 1) & 15) * 16 + (lane & 15), kc = (pi >> 5) * 64 + (pi & 1) * 32 + (lane >> 4) * 8;
                q0[e] = *(const u32x4*)(p.x + (size_t)row * KSH + kc);
            }
        }
        if (tid == 0) ctl[0] = __hip_atomic_fetch_add(wptr(myws, OFF_SEQ + (size_t)b * 128), 1u, RLX_AGENT) + 1u;
        __syncthreads();
        call = __builtin_amdgcn_readfirstlane(ctl[0]);
        char* const bufp = myws + OFF_BUF + (size_t)(call & 1) * BUF_BYTES;
        const __amdgpu_buffer_rsrc_t rx = mkrsrc(bufp, (unsigned)BUF_BYTES);
        if (g < 64) {
#pragma unroll
            for (int e = 0; e < 2; ++e)
                __builtin_amdgcn_raw_buffer_store_b128(q0[e], rx, (unsigned)((g >> 2) * BSTR + ((g >> 1) & 1) * TSTR + ((g & 1) * 16 + wid * 2 + e) * 1024 + lane * 16), 0, 17);
            WAIT_V(0);
            __syncthreads();
            publish(0, true, g >> 2, 1u);
            if (wid == 0) STAMP(17);
        }
        for (int t = g; t < NTASK; t += (g >= 64 ? NG - 64 : NTASK)) {
            int isx = 0, blk, j, stage;
            if (t < NCB1) { blk = t; j = 0; stage = 1; }
            else {
                const int t1 = t - NCB1;
                if (t1 < 3 * (16 + NCB1)) { j = 1 + t1 / (16 + NCB1); const int r = t1 % (16 + NCB1); isx = r < 16; blk = isx ? r : r - 16; stage = j == 1 ? 1 : 2; }
                else { const int r = t1 - 3 * (16 + NCB1); blk = NCB1 + (r >> 2); j = r & 3; stage = 3; }
            }
            u32x4 v[8];
            if (isx) {
#pragma unroll
                for (int e = 0; e < 8; ++e) {
                    const int pi = wid * 8 + e;
                    const int row = blk * 256 + ((pi >> 1) & 15) * 16 + (lane & 15), kc = j * 128 + (pi >> 5) * 64 + (pi & 1) * 32 + (lane >> 4) * 8;
                    v[e] = *(const u32x4*)(p.x + (size_t)row * KSH + kc);
                }
#pragma unroll
                for (int e = 0; e < 8; ++e)
                    __builtin_amdgcn_raw_buffer_store_b128(v[e], rx, (unsigned)(blk * BSTR + (2 * j + (wid >> 2)) * TSTR + ((wid & 3) * 8 + e) * 1024 + lane * 16), 0, 17);
            } else {
                const int o = wid * 2 + (lane >> 5);
                const unsigned short* src = p.w + (size_t)(j * 128 + o * 8) * N_ + blk * 256 + (lane & 31) * 8;
#pragma unroll
                for (int e = 0; e < 8; ++e) v[e] = *(const u32x4*)(src + (size_t)e * N_);
                const unsigned lo = (unsigned)((16 + blk) * BSTR + (2 * j + (o >> 3)) * TSTR + ((lane >> 4) & 1) * 8192 + ((o >> 2) & 1) * 1024 + ((o & 3) * 16 + (lane & 15)) * 16);
#pragma unroll
                for (int f = 0; f < 8; ++f) {
                    u32x4 ov;
#pragma unroll
                    for (int q = 0; q < 4; ++q)
                        ov[q] = __builtin_amdgcn_perm(v[2 * q + 1][f >> 1], v[2 * q][f >> 1], (f & 1) ? 0x07060302u : 0x05040100u);
                    __builtin_amdgcn_raw_buffer_store_b128(ov, rx, lo + (f >> 2) * 16384 + (f & 3) * 2048, 0, 17);
                }
            }
            WAIT_V(0);
            __syncthreads();
            publish(stage, isx, blk, 4u);
        }
    }
    if (wid == 0) STAMP(1);
    const unsigned par = call & 1u;
    {
        if ((g >> 3) >= GU) return;
        const int li0 = GU * ((g >> 2) & 1) + (g >> 3), rq = 4 * (g & 3);
        const int early = li0 >= NREG ? 1 : 0, ee = li0 - NREG;
        const int wr = wid >> 2, wc = wid & 3;
        const __amdgpu_buffer_rsrc_t rbuf = mkrsrc(myws + OFF_BUF + (size_t)par * BUF_BYTES, (unsigned)BUF_BYTES);
        const unsigned voff = (unsigned)(wid * 2048 + lane * 16);
        const unsigned ldsw = (unsigned)wid * 2048u;
        const int aoff = wr * 8192 + lane * 16, boff = wc * 4096 + lane * 16;
        const int ntile = early + RFULL + ((early ? ee < 2 * LEXC : li0 < LREM) ? 1 : 0);
        auto job = [&](int it) { return it < early ? ee : 4 * EC + (it - early) * 2 * GU + ((early && it - early == RFULL) ? NREG + (ee >> 1) : li0); };
        auto tile_ci = [&](int it) { return job(it) >> 2; };
        auto tile_rb = [&](int it) { return rq + (job(it) & 3); };
        auto tile_hf = [&](int it) { return (early && it - early == RFULL) ? 1 + (ee & 1) : 0; };
#define SA(bb, h) (((bb) * 2 + (h)) * 16384)
#define SB(bb, h) ((4 + (bb) * 2 + (h)) * 16384)
#define STAGE(bufoff, gofs) do { \
        __builtin_amdgcn_raw_ptr_buffer_load_lds(rbuf, (LAS void*)(lds + (bufoff) + ldsw), 16, voff, (gofs), 0, 0); \
        __builtin_amdgcn_raw_ptr_buffer_load_lds(rbuf, (LAS void*)(lds + (bufoff) + ldsw), 16, voff, (gofs), 1024, 0); } while (0)
#define LDA(dst, bb, h) do { _Pragma("unroll") for (int m = 0; m < 4; ++m) _Pragma("unroll") for (int k = 0; k < 2; ++k) dst[m][k] = *(const LAS bf16x8*)(lds + SA(bb, h) + aoff + m * 2048 + k * 1024); } while (0)
#define LDB(dst, bb, h) do { _Pragma("unroll") for (int n = 0; n < 2; ++n) _Pragma("unroll") for (int k = 0; k < 2; ++k) dst[n][k] = *(const LAS bf16x8*)(lds + SB(bb, h) + boff + n * 2048 + k * 1024); } while (0)
#define MMA(ai, bj, At, Bt) do { __builtin_amdgcn_s_setprio(1); _Pragma("unroll") for (int m = 0; m < 4; ++m) _Pragma("unroll") for (int n = 0; n < 2; ++n) _Pragma("unroll") for (int k = 0; k < 2; ++k) \
        acc[ai][bj][m][n] = __builtin_amdgcn_mfma_f32_16x16x32_bf16(Bt[n][k], At[m][k], acc[ai][bj][m][n], 0, 0, 0); __builtin_amdgcn_s_setprio(0); } while (0)

#define KSTEP(F) \
          \
        LDA(At, 0, 0); STAGE(SA(1, 1), cA + (TSTR + 16384)); \
        BAR; WAIT_L(0); MMA(0, 0, At, B0); BAR; SCHED; \
          \
        if (F) LDB(B1, 0, 1); STAGE(SB(0, 0), b2); \
        BAR; WAIT_L(0); if (F) MMA(0, 1, At, B1); BAR; \
          \
        LDA(At, 0, 1); STAGE(SA(0, 0), a2); \
        WAIT_V(10); BAR; WAIT_L(0); MMA(1, 0, At, B0); BAR; SCHED; \
          \
        LDB(B0, 1, 0); STAGE(SB(0, 1), b2 ^ 16384u); \
        WAIT_V(6); BAR; if (F) MMA(1, 1, At, B1); WAIT_L(0); BAR; SCHED; \
          \
        LDA(At, 1, 0); STAGE(SA(0, 1), a2 + 16384); \
        BAR; WAIT_L(0); MMA(0, 0, At, B0); BAR; SCHED; \
          \
        if (F) LDB(B1, 1, 1); STAGE(SB(1, 0), b2 + TSTR); \
        BAR; WAIT_L(0); if (F) MMA(0, 1, At, B1); BAR; \
          \
        LDA(At, 1, 1); STAGE(SA(1, 0), a2 + TSTR); \
        WAIT_V(10); BAR; WAIT_L(0); MMA(1, 0, At, B0); BAR; SCHED; \
          \
        LDB(B0, 0, 0); STAGE(SB(1, 1), (b2 + TSTR) ^ 16384u); \
        WAIT_V(6); BAR; if (F) MMA(1, 1, At, B1); WAIT_L(0); BAR; SCHED; \
        cA = a2; cB = b2; a2 += 2 * TSTR; b2 += 2 * TSTR;

        auto ready = [&](int rb, int cb, int seg) -> int {
            if (wid == 0) {
                const int l = lane & 15;
                const bool act = lane < 32 && l < 9;
                const unsigned need = call * (l == 0 ? 16u : (unsigned)(7 * UPI));
                unsigned* f = wptr(myws, OFF_RDY + (size_t)(((lane & 16) ? 16 + cb : rb) * 128 + l * 4));
                const unsigned long long t0 = __builtin_amdgcn_s_memrealtime();
                int nrdy;
                for (;;) {
                    const unsigned v = act ? __hip_atomic_load(f, RLX_AGENT) : need;
                    const unsigned bal = (unsigned)__ballot((int)(v - need) >= 0);
                    nrdy = __builtin_ctz(~(bal & (bal >> 16) & 0x1ffu));
                    if (nrdy > seg) break;
                    if ((unsigned)__builtin_amdgcn_readfirstlane(__hip_atomic_load(tmo, RLX_AGENT)) == call ||
                        __builtin_amdgcn_s_memrealtime() - t0 > DEADLINE) {
                        __hip_atomic_store(tmo, call, RLX_AGENT);
                        nrdy = 9; break;
                    }
                    __builtin_amdgcn_s_sleep(CFG_SLEEP);
                }
                __builtin_amdgcn_fence(__ATOMIC_ACQUIRE, "agent");
                if (lane == 0) ctl[1] = (unsigned)nrdy;
                asm volatile("s_waitcnt vmcnt(0) lgkmcnt(0)" ::: "memory");
            }
            asm volatile("" ::: "memory"); BAR; asm volatile("" ::: "memory");
            const int nr = __builtin_amdgcn_readfirstlane(*(volatile LAS unsigned*)&ctl[1]);
            WAIT_L(0);
            return 1 + 7 * nr;
        };

        int slot = 0;
        bool pre = false;
        auto prepoll = [&](int rb, int cb) {
            slot ^= 1; pre = true;
            if (wid == 0) {
                const __amdgpu_buffer_rsrc_t rc = mkrsrc(myws, (unsigned)CTL_BYTES);
                const unsigned o = (unsigned)OFF_RDY + (unsigned)(((lane & 16) ? 16 + cb : rb) * 128 + (lane & 15) * 4);
                __builtin_amdgcn_raw_ptr_buffer_load_lds(rc, (LAS void*)(lds + 131072 + 256 + slot * 256), 4, o, 0, 0, 16);
            }
        };
        auto peek = [&]() -> int {
            if (!pre) return 0;
            const unsigned v = *(volatile LAS unsigned*)(lds + 131072 + 256 + slot * 256 + lane * 4);
            const unsigned need = call * ((lane & 15) == 0 ? 16u : (unsigned)(7 * UPI));
            const unsigned bal = (unsigned)__ballot((int)(v - need) >= 0);
            return __builtin_ctz(~(bal & (bal >> 16) & 0x1ffu));
        };

        f32x4 acc[2][2][4][2];
#pragma unroll
        for (int a = 0; a < 2; ++a)
#pragma unroll
            for (int c = 0; c < 2; ++c)
#pragma unroll
                for (int m = 0; m < 4; ++m)
#pragma unroll
                    for (int n = 0; n < 2; ++n) acc[a][c][m][n] = (f32x4){0.f, 0.f, 0.f, 0.f};
        bf16x8 At[4][2], B0[2][2], B1[2][2];
        int ci = tile_ci(0), rb = tile_rb(0), cb = gcb(rank, ci);
        auto epilogue = [&](auto nbc, int r0) {
            constexpr int NB = decltype(nbc)::value;
            int tz = tid; asm volatile("" : "+v"(tz));
            const int ln = tz & 63, fr = ln & 15, fq = ln >> 4;
            const bool bad = (unsigned)__builtin_amdgcn_readfirstlane(__hip_atomic_load(tmo, RLX_AGENT)) == call;
            const bool own = ci < NO;
            const __amdgpu_buffer_rsrc_t ry = mkrsrc(myws + OFF_Y + (size_t)par * Y_BYTES, own ? (unsigned)(NO * 16 * 131072) : 0u);
            const __amdgpu_buffer_rsrc_t ro = mkrsrc(p.out, (unsigned)((size_t)M_ * N_ * 2));
            const unsigned yo = (unsigned)((ci * 16 + rb) * 131072 + wid * 16384 + ln * 16);
            const unsigned oo = (unsigned)((r0 + wc * 32 + fq * 4) * N_ + cb * 256 + wr * 128 + fr * 8) * 2u;
#pragma unroll
            for (int bj = 0; bj < NB; ++bj)
#pragma unroll
                for (int nf = 0; nf < 2; ++nf)
#pragma unroll
                    for (int j = 0; j < 4; ++j) {
                        u32x4 o;
#pragma unroll
                        for (int d = 0; d < 4; ++d)
                            o[d] = pack_bf16(silu(acc[d >> 1][bj][(d & 1) * 2][nf][j]), silu(acc[d >> 1][bj][(d & 1) * 2 + 1][nf][j]));
                        if (bad) o = (u32x4){0x7fc07fc0u, 0x7fc07fc0u, 0x7fc07fc0u, 0x7fc07fc0u};
                        __builtin_amdgcn_raw_buffer_store_b128(o, ro, oo + (unsigned)((bj * 128 + nf * 16 + j) * N_ * 2), 0, 16);
                        if (NB == 2) __builtin_amdgcn_raw_buffer_store_b128(o, ry, yo + ((bj * 2 + nf) * 4 + j) * 1024, 0, 17);
                    }
            if (NB == 2 && own) {
                WAIT_V(0);
                if (ln < 7) {
                    const unsigned peer = (unsigned)(rank + 1 + ln) & 7u;
                    __hip_atomic_fetch_add(wptr(p.ws[peer], OFF_EXP + (size_t)(rank * 128 + 16 + ci * 4)), 1u, RLX_SYS);
                }
            }
        };
        int lim = ready(rb, cb, 0);
        unsigned cA = (unsigned)(16 + cb) * BSTR, cB = (unsigned)rb * BSTR;
        STAGE(SB(0, 0), cB); STAGE(SA(0, 0), cA); STAGE(SB(0, 1), cB + 16384); STAGE(SA(0, 1), cA + 16384);
        if (wr == 1) BAR;
        WAIT_V(4); BAR;
        STAGE(SB(1, 0), cB + TSTR); STAGE(SA(1, 0), cA + TSTR); STAGE(SB(1, 1), cB + (TSTR + 16384));
        WAIT_V(6); BAR;
        LDB(B0, 0, 0);
        WAIT_L(0);
        if (wid == 0) STAMP(2);
        const int hf = tile_hf(ntile - 1);
        const int nfull = ntile - (hf ? 1 : 0);
        for (int it = 0; it < nfull; ++it) {
            const bool has_next = it + 1 < ntile;
            const int nci = has_next ? tile_ci(it + 1) : ci, nrb = has_next ? tile_rb(it + 1) : rb, ncb = gcb(rank, nci);
            const int nhf = it + 1 == nfull ? hf : 0;
            int pp = 0;
            while (pp < 32) {
                unsigned a2, b2; int run;
                if (pp < 31) {
                    if (2 * pp + 3 >= lim) {
                        const int seg = (2 * pp + 2) / 7, nr = peek();
                        lim = nr > seg ? 1 + 7 * nr : ready(rb, cb, seg);
                    }
                    pre = false;
                    run = ((lim - 2) >> 1) - pp;
                    if (run > 1) --run;
                    else if (lim < 64) prepoll(rb, cb);
                    else if (has_next) prepoll(nrb, ncb);
                    a2 = cA + 2 * TSTR; b2 = cB + 2 * TSTR;
                } else {
                    if (has_next) { const int seg = nhf ? 8 : 0, nr = peek(); lim = nr > seg ? 1 + 7 * nr : ready(nrb, ncb, seg); }
                    pre = false;
                    run = 1;
                    a2 = (unsigned)(16 + ncb) * BSTR; b2 = (unsigned)nrb * BSTR + (nhf == 2 ? 16384u : 0u);
                }
                pp += run;
                do { KSTEP(1) } while (--run > 0);
            }
            if (wid == 0) { STAMP(4 + 2 * it); STAMPC(44 + 2 * it); }
            epilogue(IC<2>{}, rb * 256);
            if (wid == 0) { STAMP(5 + 2 * it); STAMPC(45 + 2 * it); }
#pragma unroll
            for (int a = 0; a < 2; ++a)
#pragma unroll
                for (int c = 0; c < 2; ++c)
#pragma unroll
                    for (int m = 0; m < 4; ++m)
#pragma unroll
                        for (int n = 0; n < 2; ++n) acc[a][c][m][n] = (f32x4){0.f, 0.f, 0.f, 0.f};
            ci = nci; rb = nrb; cb = ncb;
        }
        if (LEXC > 0 && hf) {
            const unsigned a0 = cA, b0 = cB;
            unsigned a2 = cA + 2 * TSTR, b2 = cB + 2 * TSTR;
            for (int pp = 0; pp < 32; ++pp) {
                if (pp == 31) { a2 = a0; b2 = b0; }
                KSTEP(0)
            }
            if (wid == 0) STAMP(4 + 2 * nfull);
            epilogue(IC<1>{}, rb * 256 + (hf == 2 ? 128 : 0));
            if (wid == 0) STAMP(5 + 2 * nfull);
        }
        WAIT_V(0);
        if (wr == 0) BAR;
    }
}

}

extern "C" void kernel_launch(int rank, int nranks, ncclComm_t comm,
                              hipStream_t stream,
                              void* const* d_in, const int* in_sizes, int n_in,
                              void* d_out, int out_size,
                              const int* in_dtypes, int out_dtype,
                              void* const* d_out_all,
                              void* const* d_ws_all, size_t ws_bytes) {
    (void)comm; (void)in_sizes; (void)in_dtypes; (void)out_dtype; (void)d_out_all; (void)out_size;
    if (nranks != NRANK || n_in < 2 || ws_bytes < WS_NEED || (unsigned)rank >= (unsigned)NRANK) {
        fprintf(stderr, "unexpected configuration\n");
        return;
    }
    HIP_CHECK(hipFuncSetAttribute((const void*)fused_gemm_allgather_silu, hipFuncAttributeMaxDynamicSharedMemorySize, LDS_BYTES));
    Params p;
    p.x = (const unsigned short*)d_in[0];
    p.w = (const unsigned short*)d_in[1];
    p.out = (unsigned short*)d_out;
    for (int i = 0; i < NRANK; ++i) p.ws[i] = (char*)d_ws_all[i];
    p.rank = rank;
    hipLaunchKernelGGL(fused_gemm_allgather_silu, dim3(NWG), dim3(512), LDS_BYTES, stream, p);
    HIP_CHECK(hipGetLastError());
}
```

```cpp
#include <hip/hip_runtime.h>
#include <rccl/rccl.h>
#include <cstdio>

#define HIP_CHECK(expr) do { hipError_t _e = (expr); if (_e != hipSuccess) { \
    fprintf(stderr, "HIP error %s at %s:%d\n", hipGetErrorString(_e), __FILE__, __LINE__); \
    return; } } while (0)

namespace {

#define LAS __attribute__((address_space(3)))
typedef short bf16x8 __attribute__((ext_vector_type(8)));
typedef float f32x4 __attribute__((ext_vector_type(4)));
typedef unsigned u32x4 __attribute__((ext_vector_type(4)));
typedef unsigned u32x2 __attribute__((ext_vector_type(2)));

constexpr int NRANK = 8;
constexpr int M_ = 4096, N_ = 16384, KSH = 512;
#define CFG_NO 3
#define CFG_CPX 4
#define CFG_GU 28
#define CFG_U 8
#define CFG_D 4
#define CFG_EC 2
#define CFG_PUSH 4
#define CFG_WR 4
#define CFG_PSLEEP 2
#define CFG_FI 0
#define CFG_WPC 4
#define CFG_LAY 1
#define CFG_LDF "sc0 sc1"
#define CFG_SLEEP 16
constexpr int NO = CFG_NO;
constexpr int NREP = 64 - 8 * NO;
constexpr int NCB = NO + NREP;
constexpr int NWG = 256;
constexpr int CPX = CFG_CPX;
constexpr int NCOMMSLOT = 8 * CPX;
constexpr int NG = NWG - NCOMMSLOT;
constexpr int GU = CFG_GU;
constexpr int WPC = CFG_WPC;
constexpr int NW = CPX * WPC;
constexpr int U = CFG_U;
constexpr int D = CFG_D;
constexpr int UPI = 32 / U;
constexpr int UPT = 128 / U;
constexpr int TS = U - 2 * D;
constexpr int FI = CFG_FI;
constexpr int EC = CFG_EC;
static_assert(CPX == 3 || CPX == 4, "comm placement");
static_assert(GU <= NG / 8 && 4 * EC <= GU, "workers");
static_assert(2 * D <= U && D <= 4 && (U == 4 || U == 8), "pipeline shape");
constexpr int LLEN = 4 * (NCB - EC), RFULL = LLEN / (2 * GU), LREM = LLEN % (2 * GU), NREG = 2 * GU - 4 * EC;
constexpr int LEXC = LREM > NREG ? LREM - NREG : 0;
static_assert(2 * LEXC <= 4 * EC, "half tiles");
constexpr int rnd_end(int r) { return EC + (2 * GU * (r + 1) + 3) / 4 < NCB ? EC + (2 * GU * (r + 1) + 3) / 4 : NCB; }
constexpr int B1 = EC ? EC : rnd_end(0), B2 = rnd_end(EC ? 0 : 1), B3 = rnd_end(EC ? 1 : 2), B4 = rnd_end(EC ? 2 : 3);
static_assert(B4 == NCB && rnd_end(0) >= NO, "group count, own blocks in the first round");
constexpr int NCB1 = 8 * NO + (rnd_end(0) - NO);
constexpr int NTASK = 320 - 16;
constexpr int WR = CFG_WR, PW = 4 / WR;
constexpr unsigned CNT_S0 = 16 * WR, CNT_S1 = 2 * NCB1 + 16, CNT_S2 = 2 * NCB1 + 32, CNT_S3 = (64 - NCB1) * 4, CNT_Y = 128;
static_assert(NTASK <= 2 * NG - 64, "tasks");
constexpr int PR = CFG_PUSH;
constexpr int NIMG_IN = 128 + 8 * NCB - PR;
constexpr int NU_IN = NIMG_IN * UPI, NU = NU_IN + NO * 16 * UPT;
constexpr unsigned TPC = NU + NW;

constexpr size_t MiB = 1ull << 20;
constexpr size_t OFF_SEQ = 0;
constexpr size_t OFF_TMO = 32768;
constexpr size_t OFF_TK = 33792;
constexpr size_t OFF_EXP = 49152;
constexpr size_t OFF_RDY = 65536;
constexpr size_t CTL_BYTES = 131072;
constexpr size_t OFF_DBG = 262144;
constexpr size_t OFF_BUF = 1 * MiB;
constexpr unsigned BSTR = 32768u, TSTR = 80u * 32768u;
constexpr size_t BUF_BYTES = 160 * MiB;
constexpr size_t OFF_Y = OFF_BUF + 2 * BUF_BYTES;
constexpr size_t Y_BYTES = 16 * MiB;
constexpr size_t WS_NEED = OFF_Y + 2 * Y_BYTES;
constexpr unsigned long long DEADLINE = 100000000ull;

constexpr int LDS_BYTES = 131072 + 768;
constexpr int NST = 4 + NO;

struct Params {
    const unsigned short* x;
    const unsigned short* w;
    unsigned short* out;
    char* ws[NRANK];
    int rank;
};

#define RLX_AGENT __ATOMIC_RELAXED, __HIP_MEMORY_SCOPE_AGENT
#define RLX_SYS __ATOMIC_RELAXED, __HIP_MEMORY_SCOPE_SYSTEM
#define WAIT_V(n) asm volatile("s_waitcnt vmcnt(" #n ")" ::: "memory")
#define WAIT_L(n) asm volatile("s_waitcnt lgkmcnt(" #n ")" ::: "memory")
#define BAR __builtin_amdgcn_s_barrier()
#define SCHED __builtin_amdgcn_sched_barrier(0)

__device__ __forceinline__ unsigned* wptr(char* base, size_t off) { return (unsigned*)(base + off); }
__device__ __forceinline__ __amdgpu_buffer_rsrc_t mkrsrc(const void* p, unsigned bytes) {
    return __builtin_amdgcn_make_buffer_rsrc((void*)p, 0, bytes, 0x00020000);
}
typedef int i32x4 __attribute__((ext_vector_type(4)));
__device__ __forceinline__ i32x4 mkdesc(const void* p, unsigned bytes) {
    const unsigned long long a = (unsigned long long)p;
    i32x4 d;
    d.x = __builtin_amdgcn_readfirstlane((int)(unsigned)a);
    d.y = __builtin_amdgcn_readfirstlane((int)((unsigned)(a >> 32) & 0xffffu));
    d.z = __builtin_amdgcn_readfirstlane((int)bytes);
    d.w = 0x00020000;
    return d;
}
template <int OFF> __device__ __forceinline__ void ld_sys(u32x4& v, const i32x4& d, unsigned voff) {
    asm volatile("buffer_load_dwordx4 %0, %1, %2, 0 offen offset:%3 " CFG_LDF : "=v"(v) : "v"(voff), "s"(d), "i"(OFF) : "memory");
}
__device__ __forceinline__ void st_wt(const u32x4& v, const i32x4& d, unsigned voff) {
    asm volatile("buffer_store_dwordx4 %0, %1, %2, 0 offen sc1\n\ts_nop 1" :: "v"(v), "v"(voff), "s"(d) : "memory");
}
__device__ __forceinline__ void add_noret(const i32x4& d, unsigned voff, unsigned one) {
    asm volatile("buffer_atomic_add %0, %1, %2, 0 offen" :: "v"(one), "v"(voff), "s"(d) : "memory");
}
__device__ __forceinline__ void add_ret(unsigned& r, const i32x4& d, unsigned voff) {
    asm volatile("buffer_atomic_add %0, %1, %2, 0 offen sc0" : "+v"(r) : "v"(voff), "s"(d) : "memory");
}
template <int N> __device__ __forceinline__ void wait_for(u32x4& v) { asm volatile("s_waitcnt vmcnt(%1)" : "+v"(v) : "i"(N) : "memory"); }
__device__ __forceinline__ void ld_word(unsigned& r, const i32x4& d, unsigned voff) {
    asm volatile("buffer_load_dword %0, %1, %2, 0 offen sc0 sc1" : "=v"(r) : "v"(voff), "s"(d) : "memory");
}
constexpr int younger_ops(int e) {
    int n = 0;
    const int s0 = e - D;
    for (int s = s0; s <= e; ++s) {
        const int m = (s + U) % U;
        if (s > s0) n += (m == TS ? 2 : 0) + 1;
        if (s < e) n += 1 + (m == D ? 1 : 0);
    }
    return n;
}
constexpr int younger_tk() {
    int n = 0;
    for (int s = TS; s < U - D; ++s) n += 2 + (s == D ? 1 : 0);
    return n;
}
template <int E> struct IC { static constexpr int value = E; };
__device__ __forceinline__ int gcb(int rank, int i) { return i < NO ? rank * NO + i : 8 * NO + (i - NO); }

__device__ __forceinline__ unsigned pack_bf16(float lo, float hi) {
    unsigned r; asm("v_cvt_pk_bf16_f32 %0, %1, %2" : "=v"(r) : "v"(lo), "v"(hi)); return r;
}
__device__ __forceinline__ float silu(float y) {
    const float c = __builtin_fminf(__builtin_fmaxf(y, -60.0f), 60.0f);
    const float e = __builtin_amdgcn_exp2f(c * -1.44269504088896f);
    return y * __builtin_amdgcn_rcpf(1.0f + e);
}

#define STAMP(slot) do {} while (0)
#define STAMPEND() do {} while (0)
#define STAMPC(slot) do {} while (0)

__global__ void __launch_bounds__(512, 2) fused_gemm_allgather_silu(const Params p) {
    extern __shared__ __attribute__((aligned(16))) unsigned char shm_[];
    LAS unsigned char* lds = (LAS unsigned char*)shm_;
    LAS unsigned* ctl = (LAS unsigned*)(lds + 131072);
    const int tid = threadIdx.x, lane = tid & 63;
    const int wid = __builtin_amdgcn_readfirstlane(tid >> 6);
    const int rank = p.rank;
    const int b = blockIdx.x;
    char* const myws = p.ws[rank];
    unsigned* const tmo = wptr(myws, OFF_TMO);
    if (wid == 0) STAMP(0);

    if (b < NCOMMSLOT) {
        const int link = CPX == 4 ? 2 * (b >> 3) + (b & 1) : ((b & 7) + 3 * (b >> 3)) & 7;
        if (link >= 7 || wid >= WPC) return;
        const int src = (rank + 1 + link) & 7;
        const i32x4 dctl = mkdesc(myws, (unsigned)CTL_BYTES);
        const unsigned only0 = lane == 0 ? 0u : 0x80000000u;
        const unsigned tkoff = only0 | (unsigned)(OFF_TK + link * 128);
        unsigned tn = 1u;
        add_ret(tn, dctl, tkoff);
        const unsigned floff = lane < NST ? (unsigned)(OFF_EXP + src * 128 + lane * 4) : (lane == 15 ? (unsigned)OFF_TMO : 0x80000000u);
        unsigned fv;
        ld_word(fv, dctl, floff);
        asm volatile("s_waitcnt vmcnt(0)" : "+v"(tn), "+v"(fv) :: "memory");
        const unsigned tk0 = (unsigned)__builtin_amdgcn_readfirstlane(tn);
        const unsigned call = tk0 / TPC + 1u, par = call & 1u;
        const unsigned tbase = (call - 1u) * TPC;
        const unsigned boffs = (unsigned)(OFF_BUF + (size_t)par * BUF_BYTES);
        const unsigned yoff = (unsigned)(OFF_Y + (size_t)par * Y_BYTES);
        const i32x4 dsrc = mkdesc(p.ws[src], (unsigned)WS_NEED);
        const unsigned lane_in = (unsigned)lane * 16u, lane_out = (unsigned)((lane >> 4) * 4 * N_ + (lane & 15) * 8) * 2u;
        const unsigned one = 1u;
        const unsigned fneed = call * (lane == 0 ? CNT_S0 : lane == 1 ? CNT_S1 : lane == 2 ? CNT_S2 : lane == 3 ? CNT_S3 : CNT_Y);
        int known = 0;
        bool ok = true;
        auto update = [&]() {
            known = __builtin_ctz(~(unsigned)__ballot(lane < NST && (int)(fv - fneed) >= 0));
            if ((unsigned)__builtin_amdgcn_readlane((int)fv, 15) == call) { ok = false; known = NST; }
        };
        update();
        auto resolve = [&](unsigned u, unsigned& so, unsigned& dof, unsigned& sig) {
            int stage;
            if (u < (unsigned)NU_IN) {
                const int img = u / UPI + PR, sub = u % UPI;
                int kt, blk;
                int q = img - 8 * (16 + B1);
                blk = -1;
                if (img < 8 * (16 + B1 + FI)) {
                    kt = img / (16 + B1 + FI); const int r = img % (16 + B1 + FI);
                    if (r < 16 + B1) blk = r < 16 ? (img < 16 ? PR + (r - PR + 2 * link) % (16 - PR) : ((r + 2 * link) & 15)) : 16 + gcb(rank, (r - 16 + 2 * link) % B1);
                    q = kt * FI + (r - 16 - B1);
                }
                if (blk < 0) {
                    const int r2 = q + 8 * B1;
                    const int lo = r2 < 8 * B2 ? B1 : (r2 < 8 * B3 ? B2 : B3), hi = r2 < 8 * B2 ? B2 : (r2 < 8 * B3 ? B3 : B4);
                    const int n = hi - lo, rr = r2 - 8 * lo;
                    kt = rr / n; blk = 16 + gcb(rank, lo + (rr % n + 2 * link) % n);
                }
                stage = blk >= 16 + NCB1 ? 3 : ((blk < 16 && kt < 2) ? 0 : (kt < 4 ? 1 : 2));
                const unsigned o = boffs + (unsigned)blk * BSTR + (unsigned)sub * (U * 1024);
                so = o + (unsigned)kt * TSTR;
                dof = o + (unsigned)(8 + 7 * kt + link) * TSTR;
                sig = (unsigned)OFF_RDY + (unsigned)(blk * 128 + 4 + kt * 4);
            } else if (u < (unsigned)NU) {
                const int uy = (int)u - NU_IN, t = uy / UPT, kb = (uy % UPT) * U;
                const int ci = t >> 4, rb = (t + 2 * link) & 15, w = kb >> 4, ix = kb & 15;
                const int m = rb * 256 + (ix >> 3) * 128 + (w & 3) * 32 + ((ix >> 2) & 1) * 16;
                const int n = (src * NO + ci) * 256 + (w >> 2) * 128;
                so = yoff + (unsigned)((ci * 16 + rb) * 131072 + kb * 1024);
                dof = (unsigned)(m * N_ + n) * 2u;
                sig = 0x80000000u;
                stage = 4 + ci;
            } else { so = 0x80000000u; dof = 0x80000000u; sig = 0x80000000u; stage = -1; }
            if (known <= stage) {
                const unsigned long long t0 = __builtin_amdgcn_s_memrealtime();
                for (;;) {
                    ld_word(fv, dctl, floff);
                    asm volatile("s_waitcnt vmcnt(0)" : "+v"(fv) :: "memory");
                    update();
                    if (known > stage) break;
                    if (__builtin_amdgcn_s_memrealtime() - t0 > DEADLINE) {
                        __hip_atomic_store(tmo, call, RLX_AGENT);
                        ok = false; known = NST;
                        break;
                    }
                    __builtin_amdgcn_s_sleep(CFG_PSLEEP);
                }
            }
        };
        unsigned u = tk0 - tbase;
        unsigned so, dof, sig, psig = 0x80000000u;
        resolve(u, so, dof, sig);
        if (wid == 0) STAMP(16);
        u32x4 v[U];
        {
            const unsigned sv = so + lane_in;
            auto pro = [&](auto ec) {
                constexpr int e = decltype(ec)::value;
                if constexpr (e < D) {
                    ld_sys<e * 1024>(v[e], dsrc, sv);
                    add_noret(dctl, 0x80000000u, one);
                    if constexpr (U - D + e == D) add_noret(dctl, 0x80000000u, one);
                    static_assert(TS < U - D, "no ticket draw inside the last D steps");
                }
            };
            pro(IC<0>{}); pro(IC<1>{}); pro(IC<2>{}); pro(IC<3>{});
        }
        while (u < (unsigned)NU) {
            const bool isout = u >= (unsigned)NU_IN;
            const i32x4 ddst = mkdesc(isout ? (const void*)p.out : (const void*)myws, isout ? (unsigned)((size_t)M_ * N_ * 2) : (unsigned)WS_NEED);
            const unsigned dv = dof + (isout ? lane_out : lane_in);
            const unsigned s0 = isout ? (unsigned)N_ * 2u : 1024u, s1 = isout ? (unsigned)N_ * 32u : 4096u;
            const unsigned sv = so + lane_in, sv4 = sv + 4096u;
            const bool okc = ok;
            unsigned un = 0, nso = 0, ndof = 0, nsig = 0, nsv = 0;
            auto step = [&](auto ec) {
                constexpr int e = decltype(ec)::value;
                if constexpr (e < U) {
                    if constexpr (e == TS) { tn = 1u; add_ret(tn, dctl, tkoff); ld_word(fv, dctl, floff); }
                    if constexpr (e + D < U) {
                        ld_sys<((e + D) & 3) * 1024>(v[e + D], dsrc, e + D < 4 ? sv : sv4);
                    } else {
                        if constexpr (e + D == U) {
                            asm volatile("s_waitcnt vmcnt(%2)" : "+v"(tn), "+v"(fv) : "i"(younger_tk()) : "memory");
                            un = (unsigned)__builtin_amdgcn_readfirstlane(tn) - tbase;
                            if (ok) update();
                            resolve(un, nso, ndof, nsig);
                            nsv = nso + lane_in;
                        }
                        ld_sys<(e + D - U) * 1024>(v[e + D - U], dsrc, nsv);
                    }
                    wait_for<younger_ops(e)>(v[e]);
                    if (!okc) v[e] = (u32x4){0x7fc07fc0u, 0x7fc07fc0u, 0x7fc07fc0u, 0x7fc07fc0u};
                    st_wt(v[e], ddst, dv + (e & 3) * s0 + (e >> 2) * s1);
                    if constexpr (e == D) add_noret(dctl, only0 | psig, one);
                }
            };
            step(IC<0>{}); step(IC<1>{}); step(IC<2>{}); step(IC<3>{});
            step(IC<4>{}); step(IC<5>{}); step(IC<6>{}); step(IC<7>{});
            psig = sig;
            u = un; so = nso; dof = ndof; sig = nsig;
        }
        WAIT_V(0);
        add_noret(dctl, only0 | psig, one);
        if (wid == 0) STAMP(3);
        STAMPEND();
        return;
    }

    const int g = b - NCOMMSLOT;
    unsigned call;
    {
        char* const peerws = p.ws[(unsigned)(rank + 1 + lane) & 7u];
        auto publish = [&](int stage, bool isx, int blk, unsigned cnt) {
            if (tid < 8) {
                const size_t o = tid < 7 ? OFF_EXP + (size_t)(rank * 128 + stage * 4) : OFF_RDY + (size_t)(isx ? blk : 16 + blk) * 128;
                __hip_atomic_fetch_add(wptr(peerws, o), tid < 7 ? 1u : cnt, RLX_SYS);
            }
        };
        u32x4 q0[2][PW];
        const int s0 = g - (NG - 16 * WR);
        const int rb0 = s0 / WR, pi0 = (s0 % WR) * (32 / WR) + wid * PW;
        const unsigned s0off = (unsigned)(rb0 * BSTR + pi0 * 1024 + lane * 16);
        if (s0 >= 0) {
#pragma unroll
            for (int i = 0; i < PW; ++i) {
                const int pi = pi0 + i;
                const int row = rb0 * 256 + (pi >> 1) * 16 + (lane & 15), kc = (pi & 1) * 32 + (lane >> 4) * 8;
#pragma unroll
                for (int e = 0; e < 2; ++e) q0[e][i] = *(const u32x4*)(p.x + (size_t)row * KSH + kc + e * 64);
            }
        }
        if (tid == 0) ctl[0] = __hip_atomic_fetch_add(wptr(myws, OFF_SEQ + (size_t)b * 128), 1u, RLX_AGENT) + 1u;
        __syncthreads();
        call = __builtin_amdgcn_readfirstlane(ctl[0]);
        const size_t boffs = OFF_BUF + (size_t)(call & 1) * BUF_BYTES;
        const __amdgpu_buffer_rsrc_t rx = mkrsrc(myws + boffs, (unsigned)BUF_BYTES);
        if (s0 >= 0) {
#pragma unroll
            for (int i = 0; i < PW; ++i) {
                __builtin_amdgcn_raw_buffer_store_b128(q0[0][i], rx, s0off + i * 1024, 0, 17);
                __builtin_amdgcn_raw_buffer_store_b128(q0[1][i], rx, s0off + i * 1024 + TSTR, 0, 17);
            }
            WAIT_V(0);
            BAR; asm volatile("" ::: "memory");
            publish(0, true, rb0, (unsigned)PW);
            if (wid == 0) STAMP(17);
            if (rb0 < PR) {
#pragma unroll
                for (int k = 0; k < 7; ++k) {
                    const int l = (k + s0) % 7;
                    const __amdgpu_buffer_rsrc_t rp = mkrsrc(p.ws[(unsigned)(rank - 1 - l) & 7u] + boffs, (unsigned)BUF_BYTES);
#pragma unroll
                    for (int i = 0; i < PW; ++i)
                        __builtin_amdgcn_raw_buffer_store_b128(q0[0][i], rp, s0off + i * 1024 + (unsigned)(8 + l) * TSTR, 0, 17);
                }
                WAIT_V(0);
                BAR; asm volatile("" ::: "memory");
                if (tid < 7) __hip_atomic_fetch_add(wptr(peerws, OFF_RDY + (size_t)rb0 * 128 + 4), (unsigned)(UPI / WR), RLX_SYS);
                if (wid == 0) STAMP(18);
            }
        }
        if (s0 < 0) __builtin_amdgcn_s_sleep(48);
        for (int t = g; t < NTASK; t += NG) {
            int isx = 0, blk, j, stage;
            if (t < NCB1) { blk = t; j = 0; stage = 1; }
            else {
                const int t1 = t - NCB1;
                if (t1 < 3 * (16 + NCB1)) { j = 1 + t1 / (16 + NCB1); const int r = t1 % (16 + NCB1); isx = r < 16; blk = isx ? r : r - 16; stage = j == 1 ? 1 : 2; }
                else { const int r = t1 - 3 * (16 + NCB1); blk = NCB1 + (r >> 2); j = r & 3; stage = 3; }
            }
            u32x4 v[8];
            if (isx) {
#pragma unroll
                for (int e = 0; e < 8; ++e) {
                    const int pi = wid * 8 + e;
                    const int row = blk * 256 + ((pi >> 1) & 15) * 16 + (lane & 15), kc = j * 128 + (pi >> 5) * 64 + (pi & 1) * 32 + (lane >> 4) * 8;
                    v[e] = *(const u32x4*)(p.x + (size_t)row * KSH + kc);
                }
#pragma unroll
                for (int e = 0; e < 8; ++e)
                    __builtin_amdgcn_raw_buffer_store_b128(v[e], rx, (unsigned)(blk * BSTR + (2 * j + (wid >> 2)) * TSTR + ((wid & 3) * 8 + e) * 1024 + lane * 16), 0, 17);
            } else {
                const int o = wid * 2 + (lane >> 5);
                const unsigned short* src = p.w + (size_t)(j * 128 + o * 8) * N_ + blk * 256 + (lane & 31) * 8;
#pragma unroll
                for (int e = 0; e < 8; ++e) v[e] = *(const u32x4*)(src + (size_t)e * N_);
                const unsigned lo = (unsigned)((16 + blk) * BSTR + (2 * j + (o >> 3)) * TSTR + ((lane >> 4) & 1) * 8192 + ((o >> 2) & 1) * 1024 + ((o & 3) * 16 + (lane & 15)) * 16);
#pragma unroll
                for (int f = 0; f < 8; ++f) {
                    u32x4 ov;
#pragma unroll
                    for (int q = 0; q < 4; ++q)
                        ov[q] = __builtin_amdgcn_perm(v[2 * q + 1][f >> 1], v[2 * q][f >> 1], (f & 1) ? 0x07060302u : 0x05040100u);
                    __builtin_amdgcn_raw_buffer_store_b128(ov, rx, lo + (f >> 2) * 16384 + (f & 3) * 2048, 0, 17);
                }
            }
            WAIT_V(0);
            __syncthreads();
            publish(stage, isx, blk, 4u);
        }
    }
    if (wid == 0) STAMP(1);
    const unsigned par = call & 1u;
    {
        if ((g >> 3) >= GU) return;
        const int li0 = GU * ((g >> 2) & 1) + (g >> 3), rq = 4 * (g & 3);
        const int early = li0 >= NREG ? 1 : 0, ee = li0 - NREG;
        const int wr = wid >> 2, wc = wid & 3;
        const __amdgpu_buffer_rsrc_t rbuf = mkrsrc(myws + OFF_BUF + (size_t)par * BUF_BYTES, (unsigned)BUF_BYTES);
        const unsigned voff = (unsigned)(wid * 2048 + lane * 16);
        const unsigned ldsw = (unsigned)wid * 2048u;
        const int aoff = wr * 8192 + lane * 16, boff = wc * 4096 + lane * 16;
        const int ntile = early + RFULL + ((early ? ee < 2 * LEXC : li0 < LREM) ? 1 : 0);
        auto job = [&](int it) { return it < early ? ee : 4 * EC + (it - early) * 2 * GU + ((early && it - early == RFULL) ? NREG + (ee >> 1) : li0); };
        auto tile_ci = [&](int it) { return job(it) >> 2; };
        auto tile_rb = [&](int it) { return rq + (job(it) & 3); };
        auto tile_hf = [&](int it) { return (early && it - early == RFULL) ? 1 + (ee & 1) : 0; };
#define SA(bb, h) (((bb) * 2 + (h)) * 16384)
#define SB(bb, h) ((4 + (bb) * 2 + (h)) * 16384)
#define STAGE(bufoff, gofs) do { \
        __builtin_amdgcn_raw_ptr_buffer_load_lds(rbuf, (LAS void*)(lds + (bufoff) + ldsw), 16, voff, (gofs), 0, 0); \
        __builtin_amdgcn_raw_ptr_buffer_load_lds(rbuf, (LAS void*)(lds + (bufoff) + ldsw), 16, voff, (gofs), 1024, 0); } while (0)
#define LDA(dst, bb, h) do { _Pragma("unroll") for (int m = 0; m < 4; ++m) _Pragma("unroll") for (int k = 0; k < 2; ++k) dst[m][k] = *(const LAS bf16x8*)(lds + SA(bb, h) + aoff + m * 2048 + k * 1024); } while (0)
#define LDB(dst, bb, h) do { _Pragma("unroll") for (int n = 0; n < 2; ++n) _Pragma("unroll") for (int k = 0; k < 2; ++k) dst[n][k] = *(const LAS bf16x8*)(lds + SB(bb, h) + boff + n * 2048 + k * 1024); } while (0)
#define MMA(ai, bj, At, Bt) do { __builtin_amdgcn_s_setprio(1); _Pragma("unroll") for (int m = 0; m < 4; ++m) _Pragma("unroll") for (int n = 0; n < 2; ++n) _Pragma("unroll") for (int k = 0; k < 2; ++k) \
        acc[ai][bj][m][n] = __builtin_amdgcn_mfma_f32_16x16x32_bf16(Bt[n][k], At[m][k], acc[ai][bj][m][n], 0, 0, 0); __builtin_amdgcn_s_setprio(0); } while (0)

#define KSTEP(F) \
          \
        LDA(At, 0, 0); STAGE(SA(1, 1), cA + (TSTR + 16384)); \
        BAR; WAIT_L(0); MMA(0, 0, At, B0); BAR; SCHED; \
          \
        if (F) LDB(B1, 0, 1); STAGE(SB(0, 0), b2); \
        BAR; WAIT_L(0); if (F) MMA(0, 1, At, B1); BAR; \
          \
        LDA(At, 0, 1); STAGE(SA(0, 0), a2); \
        WAIT_V(10); BAR; WAIT_L(0); MMA(1, 0, At, B0); BAR; SCHED; \
          \
        LDB(B0, 1, 0); STAGE(SB(0, 1), b2 ^ 16384u); \
        WAIT_V(6); BAR; if (F) MMA(1, 1, At, B1); WAIT_L(0); BAR; SCHED; \
          \
        LDA(At, 1, 0); STAGE(SA(0, 1), a2 + 16384); \
        BAR; WAIT_L(0); MMA(0, 0, At, B0); BAR; SCHED; \
          \
        if (F) LDB(B1, 1, 1); STAGE(SB(1, 0), b2 + TSTR); \
        BAR; WAIT_L(0); if (F) MMA(0, 1, At, B1); BAR; \
          \
        LDA(At, 1, 1); STAGE(SA(1, 0), a2 + TSTR); \
        WAIT_V(10); BAR; WAIT_L(0); MMA(1, 0, At, B0); BAR; SCHED; \
          \
        LDB(B0, 0, 0); STAGE(SB(1, 1), (b2 + TSTR) ^ 16384u); \
        WAIT_V(6); BAR; if (F) MMA(1, 1, At, B1); WAIT_L(0); BAR; SCHED; \
        cA = a2; cB = b2; a2 += 2 * TSTR; b2 += 2 * TSTR;

        auto ready = [&](int rb, int cb, int seg) -> int {
            if (wid == 0) {
                const int l = lane & 15;
                const bool act = lane < 32 && l < 9;
                const unsigned need = call * (l == 0 ? 16u : (unsigned)(7 * UPI));
                unsigned* f = wptr(myws, OFF_RDY + (size_t)(((lane & 16) ? 16 + cb : rb) * 128 + l * 4));
                const unsigned long long t0 = __builtin_amdgcn_s_memrealtime();
                int nrdy;
                for (;;) {
                    const unsigned v = act ? __hip_atomic_load(f, RLX_AGENT) : need;
                    const unsigned bal = (unsigned)__ballot((int)(v - need) >= 0);
                    nrdy = __builtin_ctz(~(bal & (bal >> 16) & 0x1ffu));
                    if (nrdy > seg) break;
                    if ((unsigned)__builtin_amdgcn_readfirstlane(__hip_atomic_load(tmo, RLX_AGENT)) == call ||
                        __builtin_amdgcn_s_memrealtime() - t0 > DEADLINE) {
                        __hip_atomic_store(tmo, call, RLX_AGENT);
                        nrdy = 9; break;
                    }
                    __builtin_amdgcn_s_sleep(CFG_SLEEP);
                }
                __builtin_amdgcn_fence(__ATOMIC_ACQUIRE, "agent");
                if (lane == 0) ctl[1] = (unsigned)nrdy;
                asm volatile("s_waitcnt vmcnt(0) lgkmcnt(0)" ::: "memory");
            }
            asm volatile("" ::: "memory"); BAR; asm volatile("" ::: "memory");
            const int nr = __builtin_amdgcn_readfirstlane(*(volatile LAS unsigned*)&ctl[1]);
            WAIT_L(0);
            return 1 + 7 * nr;
        };

        int slot = 0;
        bool pre = false;
        auto prepoll = [&](int rb, int cb) {
            slot ^= 1; pre = true;
            if (wid == 0) {
                const __amdgpu_buffer_rsrc_t rc = mkrsrc(myws, (unsigned)CTL_BYTES);
                const unsigned o = (unsigned)OFF_RDY + (unsigned)(((lane & 16) ? 16 + cb : rb) * 128 + (lane & 15) * 4);
                __builtin_amdgcn_raw_ptr_buffer_load_lds(rc, (LAS void*)(lds + 131072 + 256 + slot * 256), 4, o, 0, 0, 16);
            }
        };
        auto peek = [&]() -> int {
            if (!pre) return 0;
            const unsigned v = *(volatile LAS unsigned*)(lds + 131072 + 256 + slot * 256 + lane * 4);
            const unsigned need = call * ((lane & 15) == 0 ? 16u : (unsigned)(7 * UPI));
            const unsigned bal = (unsigned)__ballot((int)(v - need) >= 0);
            return __builtin_ctz(~(bal & (bal >> 16) & 0x1ffu));
        };

        f32x4 acc[2][2][4][2];
#pragma unroll
        for (int a = 0; a < 2; ++a)
#pragma unroll
            for (int c = 0; c < 2; ++c)
#pragma unroll
                for (int m = 0; m < 4; ++m)
#pragma unroll
                    for (int n = 0; n < 2; ++n) acc[a][c][m][n] = (f32x4){0.f, 0.f, 0.f, 0.f};
        bf16x8 At[4][2], B0[2][2], B1[2][2];
        int ci = tile_ci(0), rb = tile_rb(0), cb = gcb(rank, ci);
        auto epilogue = [&](auto nbc, int r0) {
            constexpr int NB = decltype(nbc)::value;
            int tz = tid; asm volatile("" : "+v"(tz));
            const int ln = tz & 63, fr = ln & 15, fq = ln >> 4;
            const bool bad = (unsigned)__builtin_amdgcn_readfirstlane(__hip_atomic_load(tmo, RLX_AGENT)) == call;
            const bool own = ci < NO;
            const __amdgpu_buffer_rsrc_t ry = mkrsrc(myws + OFF_Y + (size_t)par * Y_BYTES, own ? (unsigned)(NO * 16 * 131072) : 0u);
            const __amdgpu_buffer_rsrc_t ro = mkrsrc(p.out, (unsigned)((size_t)M_ * N_ * 2));
            const unsigned yo = (unsigned)((ci * 16 + rb) * 131072 + wid * 16384 + ln * 16);
            const unsigned oo = (unsigned)((r0 + wc * 32 + fq * 4) * N_ + cb * 256 + wr * 128 + fr * 8) * 2u;
#pragma unroll
            for (int bj = 0; bj < NB; ++bj)
#pragma unroll
                for (int nf = 0; nf < 2; ++nf)
#pragma unroll
                    for (int j = 0; j < 4; ++j) {
                        u32x4 o;
#pragma unroll
                        for (int d = 0; d < 4; ++d)
                            o[d] = pack_bf16(silu(acc[d >> 1][bj][(d & 1) * 2][nf][j]), silu(acc[d >> 1][bj][(d & 1) * 2 + 1][nf][j]));
                        if (bad) o = (u32x4){0x7fc07fc0u, 0x7fc07fc0u, 0x7fc07fc0u, 0x7fc07fc0u};
                        __builtin_amdgcn_raw_buffer_store_b128(o, ro, oo + (unsigned)((bj * 128 + nf * 16 + j) * N_ * 2), 0, 16);
                        if (NB == 2) __builtin_amdgcn_raw_buffer_store_b128(o, ry, yo + ((bj * 2 + nf) * 4 + j) * 1024, 0, 17);
                    }
            if (NB == 2 && own) {
                WAIT_V(0);
                if (ln < 7) {
                    const unsigned peer = (unsigned)(rank + 1 + ln) & 7u;
                    __hip_atomic_fetch_add(wptr(p.ws[peer], OFF_EXP + (size_t)(rank * 128 + 16 + ci * 4)), 1u, RLX_SYS);
                }
            }
        };
        int lim = ready(rb, cb, 0);
        unsigned cA = (unsigned)(16 + cb) * BSTR, cB = (unsigned)rb * BSTR;
        STAGE(SB(0, 0), cB); STAGE(SA(0, 0), cA); STAGE(SB(0, 1), cB + 16384); STAGE(SA(0, 1), cA + 16384);
        if (wr == 1) BAR;
        WAIT_V(4); BAR;
        STAGE(SB(1, 0), cB + TSTR); STAGE(SA(1, 0), cA + TSTR); STAGE(SB(1, 1), cB + (TSTR + 16384));
        WAIT_V(6); BAR;
        LDB(B0, 0, 0);
        WAIT_L(0);
        if (wid == 0) STAMP(2);
        const int hf = tile_hf(ntile - 1);
        const int nfull = ntile - (hf ? 1 : 0);
        for (int it = 0; it < nfull; ++it) {
            const bool has_next = it + 1 < ntile;
            const int nci = has_next ? tile_ci(it + 1) : ci, nrb = has_next ? tile_rb(it + 1) : rb, ncb = gcb(rank, nci);
            const int nhf = it + 1 == nfull ? hf : 0;
            int pp = 0;
            while (pp < 32) {
                unsigned a2, b2; int run;
                if (pp < 31) {
                    if (2 * pp + 3 >= lim) {
                        const int seg = (2 * pp + 2) / 7, nr = peek();
                        lim = nr > seg ? 1 + 7 * nr : ready(rb, cb, seg);
                    }
                    pre = false;
                    run = ((lim - 2) >> 1) - pp;
                    if (run > 1) --run;
                    else if (lim < 64) prepoll(rb, cb);
                    else if (has_next) prepoll(nrb, ncb);
                    a2 = cA + 2 * TSTR; b2 = cB + 2 * TSTR;
                } else {
                    if (has_next) { const int seg = nhf ? 8 : 0, nr = peek(); lim = nr > seg ? 1 + 7 * nr : ready(nrb, ncb, seg); }
                    pre = false;
                    run = 1;
                    a2 = (unsigned)(16 + ncb) * BSTR; b2 = (unsigned)nrb * BSTR + (nhf == 2 ? 16384u : 0u);
                }
                pp += run;
                do { KSTEP(1) } while (--run > 0);
            }
            if (wid == 0) { STAMP(4 + 2 * it); STAMPC(44 + 2 * it); }
            epilogue(IC<2>{}, rb * 256);
            if (wid == 0) { STAMP(5 + 2 * it); STAMPC(45 + 2 * it); }
#pragma unroll
            for (int a = 0; a < 2; ++a)
#pragma unroll
                for (int c = 0; c < 2; ++c)
#pragma unroll
                    for (int m = 0; m < 4; ++m)
#pragma unroll
                        for (int n = 0; n < 2; ++n) acc[a][c][m][n] = (f32x4){0.f, 0.f, 0.f, 0.f};
            ci = nci; rb = nrb; cb = ncb;
        }
        if (LEXC > 0 && hf) {
            const unsigned a0 = cA, b0 = cB;
            unsigned a2 = cA + 2 * TSTR, b2 = cB + 2 * TSTR;
            for (int pp = 0; pp < 32; ++pp) {
                if (pp == 31) { a2 = a0; b2 = b0; }
                KSTEP(0)
            }
            if (wid == 0) STAMP(4 + 2 * nfull);
            epilogue(IC<1>{}, rb * 256 + (hf == 2 ? 128 : 0));
            if (wid == 0) STAMP(5 + 2 * nfull);
        }
        WAIT_V(0);
        if (wr == 0) BAR;
    }
}

}

extern "C" void kernel_launch(int rank, int nranks, ncclComm_t comm,
                              hipStream_t stream,
                              void* const* d_in, const int* in_sizes, int n_in,
                              void* d_out, int out_size,
                              const int* in_dtypes, int out_dtype,
                              void* const* d_out_all,
                              void* const* d_ws_all, size_t ws_bytes) {
    (void)comm; (void)in_sizes; (void)in_dtypes; (void)out_dtype; (void)d_out_all; (void)out_size;
    if (nranks != NRANK || n_in < 2 || ws_bytes < WS_NEED || (unsigned)rank >= (unsigned)NRANK) {
        fprintf(stderr, "unexpected configuration\n");
        return;
    }
    HIP_CHECK(hipFuncSetAttribute((const void*)fused_gemm_allgather_silu, hipFuncAttributeMaxDynamicSharedMemorySize, LDS_BYTES));
    Params p;
    p.x = (const unsigned short*)d_in[0];
    p.w = (const unsigned short*)d_in[1];
    p.out = (unsigned short*)d_out;
    for (int i = 0; i < NRANK; ++i) p.ws[i] = (char*)d_ws_all[i];
    p.rank = rank;
    hipLaunchKernelGGL(fused_gemm_allgather_silu, dim3(NWG), dim3(512), LDS_BYTES, stream, p);
    HIP_CHECK(hipGetLastError());
}
```

```cpp
#include <hip/hip_runtime.h>
#include <rccl/rccl.h>
#include <cstdio>

#define HIP_CHECK(expr) do { hipError_t _e = (expr); if (_e != hipSuccess) { \
    fprintf(stderr, "HIP error %s at %s:%d\n", hipGetErrorString(_e), __FILE__, __LINE__); \
    return; } } while (0)

namespace {

#define LAS __attribute__((address_space(3)))
typedef short bf16x8 __attribute__((ext_vector_type(8)));
typedef float f32x4 __attribute__((ext_vector_type(4)));
typedef unsigned u32x4 __attribute__((ext_vector_type(4)));

constexpr int NRANK = 8;
constexpr int M_ = 4096, N_ = 16384, KSH = 512;
constexpr int NO = 3;
constexpr int NREP = 64 - 8 * NO;
constexpr int NCB = NO + NREP;
constexpr int NWG = 256;
constexpr int CPX = 4;
constexpr int NCOMMSLOT = 8 * CPX;
constexpr int NG = NWG - NCOMMSLOT;
constexpr int GU = 28;
constexpr int WPC = 4;
constexpr int NW = CPX * WPC;
constexpr int U = 8;
constexpr int D = 4;
constexpr int UPI = 32 / U;
constexpr int UPT = 128 / U;
constexpr int TS = U - 2 * D;
constexpr int EC = 2;
static_assert(CPX == 3 || CPX == 4, "comm placement");
static_assert(GU <= NG / 8 && 4 * EC <= GU, "workers");
static_assert(2 * D <= U && D <= 4 && (U == 4 || U == 8), "pipeline shape");
constexpr int LLEN = 4 * (NCB - EC), RFULL = LLEN / (2 * GU), LREM = LLEN % (2 * GU), NREG = 2 * GU - 4 * EC;
constexpr int LEXC = LREM > NREG ? LREM - NREG : 0;
static_assert(2 * LEXC <= 4 * EC, "half tiles");
constexpr int rnd_end(int r) { return EC + (2 * GU * (r + 1) + 3) / 4 < NCB ? EC + (2 * GU * (r + 1) + 3) / 4 : NCB; }
constexpr int GB1 = EC ? EC : rnd_end(0), GB2 = rnd_end(EC ? 0 : 1), GB3 = rnd_end(EC ? 1 : 2), GB4 = rnd_end(EC ? 2 : 3);
static_assert(GB4 == NCB && rnd_end(0) >= NO, "group count, own blocks in the first round");
constexpr int NCB1 = 8 * NO + (rnd_end(0) - NO);
constexpr int NTASK = 320 - 16;
constexpr int WR = 4, PW = 4 / WR;
constexpr unsigned CNT_S0 = 16 * WR, CNT_S1 = 2 * NCB1 + 16, CNT_S2 = 2 * NCB1 + 32, CNT_S3 = (64 - NCB1) * 4, CNT_Y = 128;
static_assert(NTASK <= 2 * NG - 64, "tasks");
constexpr int PR = 4;
constexpr int NIMG_IN = 128 + 8 * NCB - PR;
constexpr int NU_IN = NIMG_IN * UPI, NU = NU_IN + NO * 16 * UPT;
constexpr unsigned TPC = NU + NW;

constexpr size_t MiB = 1ull << 20;
constexpr size_t OFF_SEQ = 0;
constexpr size_t OFF_TMO = 32768;
constexpr size_t OFF_TK = 33792;
constexpr size_t OFF_EXP = 49152;
constexpr size_t OFF_RDY = 65536;
constexpr size_t CTL_BYTES = 131072;
constexpr size_t OFF_BUF = 1 * MiB;
constexpr unsigned BSTR = 32768u, TSTR = 80u * 32768u;
constexpr size_t BUF_BYTES = 160 * MiB;
constexpr size_t OFF_Y = OFF_BUF + 2 * BUF_BYTES;
constexpr size_t Y_BYTES = 16 * MiB;
constexpr size_t WS_NEED = OFF_Y + 2 * Y_BYTES;
constexpr unsigned long long DEADLINE = 100000000ull;

constexpr int LDS_BYTES = 131072 + 768;
constexpr int NST = 4 + NO;

struct Params {
    const unsigned short* x;
    const unsigned short* w;
    unsigned short* out;
    char* ws[NRANK];
    int rank;
};

#define RLX_AGENT __ATOMIC_RELAXED, __HIP_MEMORY_SCOPE_AGENT
#define RLX_SYS __ATOMIC_RELAXED, __HIP_MEMORY_SCOPE_SYSTEM
#define WAIT_V(n) asm volatile("s_waitcnt vmcnt(" #n ")" ::: "memory")
#define WAIT_L(n) asm volatile("s_waitcnt lgkmcnt(" #n ")" ::: "memory")
#define BAR __builtin_amdgcn_s_barrier()
#define SCHED __builtin_amdgcn_sched_barrier(0)

__device__ __forceinline__ unsigned* wptr(char* base, size_t off) { return (unsigned*)(base + off); }
__device__ __forceinline__ __amdgpu_buffer_rsrc_t mkrsrc(const void* p, unsigned bytes) {
    return __builtin_amdgcn_make_buffer_rsrc((void*)p, 0, bytes, 0x00020000);
}
typedef int i32x4 __attribute__((ext_vector_type(4)));
__device__ __forceinline__ i32x4 mkdesc(const void* p, unsigned bytes) {
    const unsigned long long a = (unsigned long long)p;
    i32x4 d;
    d.x = __builtin_amdgcn_readfirstlane((int)(unsigned)a);
    d.y = __builtin_amdgcn_readfirstlane((int)((unsigned)(a >> 32) & 0xffffu));
    d.z = __builtin_amdgcn_readfirstlane((int)bytes);
    d.w = 0x00020000;
    return d;
}
template <int OFF> __device__ __forceinline__ void ld_sys(u32x4& v, const i32x4& d, unsigned voff) {
    asm volatile("buffer_load_dwordx4 %0, %1, %2, 0 offen offset:%3 sc0 sc1" : "=v"(v) : "v"(voff), "s"(d), "i"(OFF) : "memory");
}
__device__ __forceinline__ void st_wt(const u32x4& v, const i32x4& d, unsigned voff) {
    asm volatile("buffer_store_dwordx4 %0, %1, %2, 0 offen sc1\n\ts_nop 1" :: "v"(v), "v"(voff), "s"(d) : "memory");
}
__device__ __forceinline__ void add_noret(const i32x4& d, unsigned voff, unsigned one) {
    asm volatile("buffer_atomic_add %0, %1, %2, 0 offen" :: "v"(one), "v"(voff), "s"(d) : "memory");
}
__device__ __forceinline__ void add_ret(unsigned& r, const i32x4& d, unsigned voff) {
    asm volatile("buffer_atomic_add %0, %1, %2, 0 offen sc0" : "+v"(r) : "v"(voff), "s"(d) : "memory");
}
template <int N> __device__ __forceinline__ void wait_for(u32x4& v) { asm volatile("s_waitcnt vmcnt(%1)" : "+v"(v) : "i"(N) : "memory"); }
__device__ __forceinline__ void ld_word(unsigned& r, const i32x4& d, unsigned voff) {
    asm volatile("buffer_load_dword %0, %1, %2, 0 offen sc0 sc1" : "=v"(r) : "v"(voff), "s"(d) : "memory");
}
constexpr int younger_ops(int e) {
    int n = 0;
    const int s0 = e - D;
    for (int s = s0; s <= e; ++s) {
        const int m = (s + U) % U;
        if (s > s0) n += (m == TS ? 2 : 0) + 1;
        if (s < e) n += 1 + (m == D ? 1 : 0);
    }
    return n;
}
constexpr int younger_tk() {
    int n = 0;
    for (int s = TS; s < U - D; ++s) n += 2 + (s == D ? 1 : 0);
    return n;
}
template <int E> struct IC { static constexpr int value = E; };
__device__ __forceinline__ int gcb(int rank, int i) { return i < NO ? rank * NO + i : 8 * NO + (i - NO); }

__device__ __forceinline__ unsigned pack_bf16(float lo, float hi) {
    unsigned r; asm("v_cvt_pk_bf16_f32 %0, %1, %2" : "=v"(r) : "v"(lo), "v"(hi)); return r;
}
__device__ __forceinline__ f32x4 silu4(f32x4 y) {
    const f32x4 t = y * -1.44269504088896f;
    f32x4 e, r;
#pragma unroll
    for (int i = 0; i < 4; ++i) e[i] = __builtin_amdgcn_exp2f(__builtin_amdgcn_fmed3f(t[i], -86.5617024533f, 86.5617024533f));
    const f32x4 d = e + 1.0f;
#pragma unroll
    for (int i = 0; i < 4; ++i) r[i] = __builtin_amdgcn_rcpf(d[i]);
    return y * r;
}


__global__ void __launch_bounds__(512, 2) fused_gemm_allgather_silu(const Params p) {
    extern __shared__ __attribute__((aligned(16))) unsigned char shm_[];
    LAS unsigned char* lds = (LAS unsigned char*)shm_;
    LAS unsigned* ctl = (LAS unsigned*)(lds + 131072);
    const int tid = threadIdx.x, lane = tid & 63;
    const int wid = __builtin_amdgcn_readfirstlane(tid >> 6);
    const int rank = p.rank;
    const int b = blockIdx.x;
    char* const myws = p.ws[rank];
    unsigned* const tmo = wptr(myws, OFF_TMO);

    if (b < NCOMMSLOT) {
        const int link = CPX == 4 ? 2 * (b >> 3) + (b & 1) : ((b & 7) + 3 * (b >> 3)) & 7;
        if (link >= 7 || wid >= WPC) return;
        const int src = (rank + 1 + link) & 7;
        const i32x4 dctl = mkdesc(myws, (unsigned)CTL_BYTES);
        const unsigned only0 = lane == 0 ? 0u : 0x80000000u;
        const unsigned tkoff = only0 | (unsigned)(OFF_TK + link * 128);
        const unsigned floff = lane < NST ? (unsigned)(OFF_EXP + src * 128 + lane * 4) : (lane == 15 ? (unsigned)OFF_TMO : 0x80000000u);
        const unsigned fcnt = lane == 0 ? CNT_S0 : lane == 1 ? CNT_S1 : lane == 2 ? CNT_S2 : lane == 3 ? CNT_S3 : CNT_Y;
        const i32x4 dsrc = mkdesc(p.ws[src], (unsigned)WS_NEED);
        const unsigned lane_in = (unsigned)lane * 16u, lane_out = (unsigned)((lane >> 4) * 4 * N_ + (lane & 15) * 8) * 2u;
        const unsigned one = 1u;
        unsigned tn, fv, call = 0, boffs = 0, yoff = 0;
        int known = 0;
        bool ok = true;
        auto update = [&]() {
            known = __builtin_ctz(~(unsigned)__ballot(lane < NST && (int)(fv - call * fcnt) >= 0));
            if ((unsigned)__builtin_amdgcn_readlane((int)fv, 15) == call) { ok = false; known = NST; }
        };
        auto resolve = [&](unsigned u, unsigned& so, unsigned& dof, unsigned& sig) {
            int stage;
            if (u < (unsigned)NU_IN) {
                const int img = u / UPI + PR, sub = u % UPI;
                int kt, blk;
                if (img < 8 * (16 + GB1)) {
                    kt = img / (16 + GB1); const int r = img % (16 + GB1);
                    blk = r < 16 ? (img < 16 ? PR + (r - PR + 2 * link) % (16 - PR) : ((r + 2 * link) & 15)) : 16 + gcb(rank, (r - 16 + 2 * link) % GB1);
                } else {
                    const int r2 = img - 128;
                    const int lo = r2 < 8 * GB2 ? GB1 : (r2 < 8 * GB3 ? GB2 : GB3), hi = r2 < 8 * GB2 ? GB2 : (r2 < 8 * GB3 ? GB3 : GB4);
                    const int n = hi - lo, rr = r2 - 8 * lo;
                    kt = rr / n; blk = 16 + gcb(rank, lo + (rr % n + 2 * link) % n);
                }
                stage = blk >= 16 + NCB1 ? 3 : ((blk < 16 && kt < 2) ? 0 : (kt < 4 ? 1 : 2));
                const unsigned o = boffs + (unsigned)blk * BSTR + (unsigned)sub * (U * 1024);
                so = o + (unsigned)kt * TSTR;
                dof = o + (unsigned)(8 + 7 * kt + link) * TSTR;
                sig = (unsigned)OFF_RDY + (unsigned)(blk * 128 + 4 + kt * 4);
            } else if (u < (unsigned)NU) {
                const int uy = (int)u - NU_IN, t = uy / UPT, kb = (uy % UPT) * U;
                const int ci = t >> 4, rb = (t + 2 * link) & 15, w = kb >> 4, ix = kb & 15;
                const int m = rb * 256 + (ix >> 3) * 128 + (w & 3) * 32 + ((ix >> 2) & 1) * 16;
                const int n = (src * NO + ci) * 256 + (w >> 2) * 128;
                so = yoff + (unsigned)((ci * 16 + rb) * 131072 + kb * 1024);
                dof = (unsigned)(m * N_ + n) * 2u;
                sig = 0x80000000u;
                stage = 4 + ci;
            } else { so = 0x80000000u; dof = 0x80000000u; sig = 0x80000000u; stage = -1; }
            if (known <= stage) {
                const unsigned long long t0 = __builtin_amdgcn_s_memrealtime();
                for (;;) {
                    ld_word(fv, dctl, floff);
                    asm volatile("s_waitcnt vmcnt(0)" : "+v"(fv) :: "memory");
                    update();
                    if (known > stage) break;
                    if (__builtin_amdgcn_s_memrealtime() - t0 > DEADLINE) {
                        __hip_atomic_store(tmo, call, RLX_AGENT);
                        ok = false; known = NST;
                        break;
                    }
                    __builtin_amdgcn_s_sleep(2);
                }
            }
        };
        unsigned u = 0, so = 0x80000000u, dof = 0x80000000u, sig = 0x80000000u, psig = 0x80000000u;
        u32x4 v[U];
#pragma unroll
        for (int e = 0; e < D; ++e) v[e] = (u32x4){0u, 0u, 0u, 0u};
        while (u < (unsigned)NU) {
            const bool isout = u >= (unsigned)NU_IN;
            const i32x4 ddst = mkdesc(isout ? (const void*)p.out : (const void*)myws, isout ? (unsigned)((size_t)M_ * N_ * 2) : (unsigned)WS_NEED);
            const unsigned dv = dof + (isout ? lane_out : lane_in);
            const unsigned s0 = isout ? (unsigned)N_ * 2u : 1024u, s1 = isout ? (unsigned)N_ * 32u : 4096u;
            const unsigned sv = so + lane_in, sv4 = sv + 4096u;
            const bool okc = ok;
            unsigned un = 0, nso = 0, ndof = 0, nsig = 0, nsv = 0;
            auto step = [&](auto ec) {
                constexpr int e = decltype(ec)::value;
                if constexpr (e < U) {
                    if constexpr (e == TS) { tn = 1u; add_ret(tn, dctl, tkoff); ld_word(fv, dctl, floff); }
                    if constexpr (e + D < U) {
                        ld_sys<((e + D) & 3) * 1024>(v[e + D], dsrc, e + D < 4 ? sv : sv4);
                    } else {
                        if constexpr (e + D == U) {
                            asm volatile("s_waitcnt vmcnt(%2)" : "+v"(tn), "+v"(fv) : "i"(younger_tk()) : "memory");
                            const unsigned tk = (unsigned)__builtin_amdgcn_readfirstlane(tn);
                            call = tk / TPC + 1u;
                            boffs = (unsigned)OFF_BUF + (call & 1u) * (unsigned)BUF_BYTES;
                            yoff = (unsigned)OFF_Y + (call & 1u) * (unsigned)Y_BYTES;
                            un = tk - (call - 1u) * TPC;
                            if (ok) update();
                            resolve(un, nso, ndof, nsig);
                            nsv = nso + lane_in;
                        }
                        ld_sys<(e + D - U) * 1024>(v[e + D - U], dsrc, nsv);
                    }
                    wait_for<younger_ops(e)>(v[e]);
                    if (!okc) v[e] = (u32x4){0x7fc07fc0u, 0x7fc07fc0u, 0x7fc07fc0u, 0x7fc07fc0u};
                    st_wt(v[e], ddst, dv + (e & 3) * s0 + (e >> 2) * s1);
                    if constexpr (e == D) add_noret(dctl, only0 | psig, one);
                }
            };
            step(IC<0>{}); step(IC<1>{}); step(IC<2>{}); step(IC<3>{});
            step(IC<4>{}); step(IC<5>{}); step(IC<6>{}); step(IC<7>{});
            psig = sig;
            u = un; so = nso; dof = ndof; sig = nsig;
        }
        WAIT_V(0);
        add_noret(dctl, only0 | psig, one);
        return;
    }

    const int g = b - NCOMMSLOT;
    unsigned call;
    {
        char* const peerws = p.ws[(unsigned)(rank + 1 + lane) & 7u];
        auto publish = [&](int stage, bool isx, int blk, unsigned cnt) {
            if (tid < 8) {
                const size_t o = tid < 7 ? OFF_EXP + (size_t)(rank * 128 + stage * 4) : OFF_RDY + (size_t)(isx ? blk : 16 + blk) * 128;
                __hip_atomic_fetch_add(wptr(peerws, o), tid < 7 ? 1u : cnt, RLX_SYS);
            }
        };
        u32x4 q0[2][PW];
        const int s0 = g - (NG - 16 * WR);
        const int rb0 = s0 / WR, pi0 = (s0 % WR) * (32 / WR) + wid * PW;
        const unsigned s0off = (unsigned)(rb0 * BSTR + pi0 * 1024 + lane * 16);
        if (s0 >= 0) {
#pragma unroll
            for (int i = 0; i < PW; ++i) {
                const int pi = pi0 + i;
                const int row = rb0 * 256 + (pi >> 1) * 16 + (lane & 15), kc = (pi & 1) * 32 + (lane >> 4) * 8;
#pragma unroll
                for (int e = 0; e < 2; ++e) q0[e][i] = *(const u32x4*)(p.x + (size_t)row * KSH + kc + e * 64);
            }
        }
        if (tid == 0) ctl[0] = __hip_atomic_fetch_add(wptr(myws, OFF_SEQ + (size_t)b * 128), 1u, RLX_AGENT) + 1u;
        __syncthreads();
        call = __builtin_amdgcn_readfirstlane(ctl[0]);
        const size_t boffs = OFF_BUF + (size_t)(call & 1) * BUF_BYTES;
        const __amdgpu_buffer_rsrc_t rx = mkrsrc(myws + boffs, (unsigned)BUF_BYTES);
        if (s0 >= 0) {
#pragma unroll
            for (int i = 0; i < PW; ++i) {
                __builtin_amdgcn_raw_buffer_store_b128(q0[0][i], rx, s0off + i * 1024, 0, 17);
                __builtin_amdgcn_raw_buffer_store_b128(q0[1][i], rx, s0off + i * 1024 + TSTR, 0, 17);
            }
            WAIT_V(0);
            BAR; asm volatile("" ::: "memory");
            publish(0, true, rb0, (unsigned)PW);
            if (rb0 < PR) {
#pragma unroll
                for (int k = 0; k < 7; ++k) {
                    const int l = (k + s0) % 7;
                    const __amdgpu_buffer_rsrc_t rp = mkrsrc(p.ws[(unsigned)(rank - 1 - l) & 7u] + boffs, (unsigned)BUF_BYTES);
#pragma unroll
                    for (int i = 0; i < PW; ++i)
                        __builtin_amdgcn_raw_buffer_store_b128(q0[0][i], rp, s0off + i * 1024 + (unsigned)(8 + l) * TSTR, 0, 17);
                }
                WAIT_V(0);
                BAR; asm volatile("" ::: "memory");
                if (tid < 7) __hip_atomic_fetch_add(wptr(peerws, OFF_RDY + (size_t)rb0 * 128 + 4), (unsigned)(UPI / WR), RLX_SYS);
            }
        }
        if (s0 < 0) __builtin_amdgcn_s_sleep(48);
        for (int t = g; t < NTASK; t += NG) {
            int isx = 0, blk, j, stage;
            if (t < NCB1) { blk = t; j = 0; stage = 1; }
            else {
                const int t1 = t - NCB1;
                if (t1 < 3 * (16 + NCB1)) { j = 1 + t1 / (16 + NCB1); const int r = t1 % (16 + NCB1); isx = r < 16; blk = isx ? r : r - 16; stage = j == 1 ? 1 : 2; }
                else { const int r = t1 - 3 * (16 + NCB1); blk = NCB1 + (r >> 2); j = r & 3; stage = 3; }
            }
            u32x4 v[8];
            if (isx) {
#pragma unroll
                for (int e = 0; e < 8; ++e) {
                    const int pi = wid * 8 + e;
                    const int row = blk * 256 + ((pi >> 1) & 15) * 16 + (lane & 15), kc = j * 128 + (pi >> 5) * 64 + (pi & 1) * 32 + (lane >> 4) * 8;
                    v[e] = *(const u32x4*)(p.x + (size_t)row * KSH + kc);
                }
#pragma unroll
                for (int e = 0; e < 8; ++e)
                    __builtin_amdgcn_raw_buffer_store_b128(v[e], rx, (unsigned)(blk * BSTR + (2 * j + (wid >> 2)) * TSTR + ((wid & 3) * 8 + e) * 1024 + lane * 16), 0, 17);
            } else {
                const int o = wid * 2 + (lane >> 5);
                const unsigned short* src = p.w + (size_t)(j * 128 + o * 8) * N_ + blk * 256 + (lane & 31) * 8;
#pragma unroll
                for (int e = 0; e < 8; ++e) v[e] = *(const u32x4*)(src + (size_t)e * N_);
                const unsigned lo = (unsigned)((16 + blk) * BSTR + (2 * j + (o >> 3)) * TSTR + ((lane >> 4) & 1) * 8192 + ((o >> 2) & 1) * 1024 + ((o & 3) * 16 + (lane & 15)) * 16);
#pragma unroll
                for (int f = 0; f < 8; ++f) {
                    u32x4 ov;
#pragma unroll
                    for (int q = 0; q < 4; ++q)
                        ov[q] = __builtin_amdgcn_perm(v[2 * q + 1][f >> 1], v[2 * q][f >> 1], (f & 1) ? 0x07060302u : 0x05040100u);
                    __builtin_amdgcn_raw_buffer_store_b128(ov, rx, lo + (f >> 2) * 16384 + (f & 3) * 2048, 0, 17);
                }
            }
            WAIT_V(0);
            __syncthreads();
            publish(stage, isx, blk, 4u);
        }
    }
    const unsigned par = call & 1u;
    {
        if ((g >> 3) >= GU) return;
        const int li0 = GU * ((g >> 2) & 1) + (g >> 3), rq = 4 * (g & 3);
        const int early = li0 >= NREG ? 1 : 0, ee = li0 - NREG;
        const int wr = wid >> 2, wc = wid & 3;
        const __amdgpu_buffer_rsrc_t rbuf = mkrsrc(myws + OFF_BUF + (size_t)par * BUF_BYTES, (unsigned)BUF_BYTES);
        const unsigned voff = (unsigned)(wid * 2048 + lane * 16);
        const unsigned ldsw = (unsigned)wid * 2048u;
        const int aoff = wr * 8192 + lane * 16, boff = wc * 4096 + lane * 16;
        const int ntile = early + RFULL + ((early ? ee < 2 * LEXC : li0 < LREM) ? 1 : 0);
        auto job = [&](int it) { return it < early ? ee : 4 * EC + (it - early) * 2 * GU + ((early && it - early == RFULL) ? NREG + (ee >> 1) : li0); };
        auto tile_ci = [&](int it) { return job(it) >> 2; };
        auto tile_rb = [&](int it) { return rq + (job(it) & 3); };
        auto tile_hf = [&](int it) { return (early && it - early == RFULL) ? 1 + (ee & 1) : 0; };
#define SA(bb, h) (((bb) * 2 + (h)) * 16384)
#define SB(bb, h) ((4 + (bb) * 2 + (h)) * 16384)
#define STAGE(bufoff, gofs) do { \
        __builtin_amdgcn_raw_ptr_buffer_load_lds(rbuf, (LAS void*)(lds + (bufoff) + ldsw), 16, voff, (gofs), 0, 0); \
        __builtin_amdgcn_raw_ptr_buffer_load_lds(rbuf, (LAS void*)(lds + (bufoff) + ldsw), 16, voff, (gofs), 1024, 0); } while (0)
#define LDA(dst, bb, h) do { _Pragma("unroll") for (int m = 0; m < 4; ++m) _Pragma("unroll") for (int k = 0; k < 2; ++k) dst[m][k] = *(const LAS bf16x8*)(lds + SA(bb, h) + aoff + m * 2048 + k * 1024); } while (0)
#define LDB(dst, bb, h) do { _Pragma("unroll") for (int n = 0; n < 2; ++n) _Pragma("unroll") for (int k = 0; k < 2; ++k) dst[n][k] = *(const LAS bf16x8*)(lds + SB(bb, h) + boff + n * 2048 + k * 1024); } while (0)
#define MMA(ai, bj, At, Bt) do { __builtin_amdgcn_s_setprio(1); _Pragma("unroll") for (int m = 0; m < 4; ++m) _Pragma("unroll") for (int n = 0; n < 2; ++n) _Pragma("unroll") for (int k = 0; k < 2; ++k) \
        acc[ai][bj][m][n] = __builtin_amdgcn_mfma_f32_16x16x32_bf16(Bt[n][k], At[m][k], acc[ai][bj][m][n], 0, 0, 0); __builtin_amdgcn_s_setprio(0); } while (0)

#define KSTEP(F) \
          \
        LDA(At, 0, 0); STAGE(SA(1, 1), cA + (TSTR + 16384)); \
        BAR; WAIT_L(0); MMA(0, 0, At, B0); BAR; SCHED; \
          \
        if (F) LDB(B1, 0, 1); STAGE(SB(0, 0), b2); \
        BAR; WAIT_L(0); if (F) MMA(0, 1, At, B1); BAR; \
          \
        LDA(At, 0, 1); STAGE(SA(0, 0), a2); \
        WAIT_V(10); BAR; WAIT_L(0); MMA(1, 0, At, B0); BAR; SCHED; \
          \
        LDB(B0, 1, 0); STAGE(SB(0, 1), b2 ^ 16384u); \
        WAIT_V(6); BAR; if (F) MMA(1, 1, At, B1); WAIT_L(0); BAR; SCHED; \
          \
        LDA(At, 1, 0); STAGE(SA(0, 1), a2 + 16384); \
        BAR; WAIT_L(0); MMA(0, 0, At, B0); BAR; SCHED; \
          \
        if (F) LDB(B1, 1, 1); STAGE(SB(1, 0), b2 + TSTR); \
        BAR; WAIT_L(0); if (F) MMA(0, 1, At, B1); BAR; \
          \
        LDA(At, 1, 1); STAGE(SA(1, 0), a2 + TSTR); \
        WAIT_V(10); BAR; WAIT_L(0); MMA(1, 0, At, B0); BAR; SCHED; \
          \
        LDB(B0, 0, 0); STAGE(SB(1, 1), (b2 + TSTR) ^ 16384u); \
        WAIT_V(6); BAR; if (F) MMA(1, 1, At, B1); WAIT_L(0); BAR; SCHED; \
        cA = a2; cB = b2; a2 += 2 * TSTR; b2 += 2 * TSTR;

        auto ready = [&](int rb, int cb, int seg) -> int {
            if (wid == 0) {
                const int l = lane & 15;
                const bool act = lane < 32 && l < 9;
                const unsigned need = call * (l == 0 ? 16u : (unsigned)(7 * UPI));
                unsigned* f = wptr(myws, OFF_RDY + (size_t)(((lane & 16) ? 16 + cb : rb) * 128 + l * 4));
                const unsigned long long t0 = __builtin_amdgcn_s_memrealtime();
                int nrdy;
                for (;;) {
                    const unsigned v = act ? __hip_atomic_load(f, RLX_SYS) : need;
                    const unsigned bal = (unsigned)__ballot((int)(v - need) >= 0);
                    nrdy = __builtin_ctz(~(bal & (bal >> 16) & 0x1ffu));
                    if (nrdy > seg) break;
                    if ((unsigned)__builtin_amdgcn_readfirstlane(__hip_atomic_load(tmo, RLX_AGENT)) == call ||
                        __builtin_amdgcn_s_memrealtime() - t0 > DEADLINE) {
                        __hip_atomic_store(tmo, call, RLX_AGENT);
                        nrdy = 9; break;
                    }
                    __builtin_amdgcn_s_sleep(16);
                }
                __builtin_amdgcn_fence(__ATOMIC_ACQUIRE, "agent");
                if (lane == 0) ctl[1] = (unsigned)nrdy;
                asm volatile("s_waitcnt vmcnt(0) lgkmcnt(0)" ::: "memory");
            }
            asm volatile("" ::: "memory"); BAR; asm volatile("" ::: "memory");
            const int nr = __builtin_amdgcn_readfirstlane(*(volatile LAS unsigned*)&ctl[1]);
            WAIT_L(0);
            return 1 + 7 * nr;
        };

        int slot = 0;
        bool pre = false;
        auto prepoll = [&](int rb, int cb) {
            slot ^= 1; pre = true;
            if (wid == 0) {
                const __amdgpu_buffer_rsrc_t rc = mkrsrc(myws, (unsigned)CTL_BYTES);
                const unsigned o = (unsigned)OFF_RDY + (unsigned)(((lane & 16) ? 16 + cb : rb) * 128 + (lane & 15) * 4);
                __builtin_amdgcn_raw_ptr_buffer_load_lds(rc, (LAS void*)(lds + 131072 + 256 + slot * 256), 4, o, 0, 0, 17);
            }
        };
        auto peek = [&]() -> int {
            if (!pre) return 0;
            const unsigned v = *(volatile LAS unsigned*)(lds + 131072 + 256 + slot * 256 + lane * 4);
            const unsigned need = call * ((lane & 15) == 0 ? 16u : (unsigned)(7 * UPI));
            const unsigned bal = (unsigned)__ballot((int)(v - need) >= 0);
            return __builtin_ctz(~(bal & (bal >> 16) & 0x1ffu));
        };

        f32x4 acc[2][2][4][2];
#pragma unroll
        for (int a = 0; a < 2; ++a)
#pragma unroll
            for (int c = 0; c < 2; ++c)
#pragma unroll
                for (int m = 0; m < 4; ++m)
#pragma unroll
                    for (int n = 0; n < 2; ++n) acc[a][c][m][n] = (f32x4){0.f, 0.f, 0.f, 0.f};
        bf16x8 At[4][2], B0[2][2], B1[2][2];
        int ci = tile_ci(0), rb = tile_rb(0), cb = gcb(rank, ci);
        auto epilogue = [&](auto nbc, int r0) {
            constexpr int NB = decltype(nbc)::value;
            int tz = tid; asm volatile("" : "+v"(tz));
            const int ln = tz & 63, fr = ln & 15, fq = ln >> 4;
            const unsigned nanm = (unsigned)__builtin_amdgcn_readfirstlane(__hip_atomic_load(tmo, RLX_AGENT)) == call ? 0x7fc07fc0u : 0u;
            const bool own = ci < NO;
            const __amdgpu_buffer_rsrc_t ry = mkrsrc(myws + OFF_Y + (size_t)par * Y_BYTES, own ? (unsigned)(NO * 16 * 131072) : 0u);
            const __amdgpu_buffer_rsrc_t ro = mkrsrc(p.out, (unsigned)((size_t)M_ * N_ * 2));
            const unsigned yo = (unsigned)((ci * 16 + rb) * 131072 + wid * 16384 + ln * 16);
            const unsigned oo = (unsigned)((r0 + wc * 32 + fq * 4) * N_ + cb * 256 + wr * 128 + fr * 8) * 2u;
#pragma unroll
            for (int bj = 0; bj < NB; ++bj)
#pragma unroll
                for (int nf = 0; nf < 2; ++nf) {
                    f32x4 sv[8];
#pragma unroll
                    for (int f = 0; f < 8; ++f) sv[f] = silu4(acc[f >> 2][bj][f & 3][nf]);
#pragma unroll
                    for (int j = 0; j < 4; ++j) {
                        u32x4 o;
#pragma unroll
                        for (int d = 0; d < 4; ++d) o[d] = pack_bf16(sv[2 * d][j], sv[2 * d + 1][j]) | nanm;
                        __builtin_amdgcn_raw_buffer_store_b128(o, ro, oo + (unsigned)((bj * 128 + nf * 16 + j) * N_ * 2), 0, 16);
                        if (NB == 2) __builtin_amdgcn_raw_buffer_store_b128(o, ry, yo + ((bj * 2 + nf) * 4 + j) * 1024, 0, 17);
                    }
                }
            if (NB == 2 && own) {
                WAIT_V(0);
                if (ln < 7) {
                    const unsigned peer = (unsigned)(rank + 1 + ln) & 7u;
                    __hip_atomic_fetch_add(wptr(p.ws[peer], OFF_EXP + (size_t)(rank * 128 + 16 + ci * 4)), 1u, RLX_SYS);
                }
            }
        };
        int lim = ready(rb, cb, 0);
        unsigned cA = (unsigned)(16 + cb) * BSTR, cB = (unsigned)rb * BSTR;
        STAGE(SB(0, 0), cB); STAGE(SA(0, 0), cA); STAGE(SB(0, 1), cB + 16384); STAGE(SA(0, 1), cA + 16384);
        if (wr == 1) BAR;
        WAIT_V(4); BAR;
        STAGE(SB(1, 0), cB + TSTR); STAGE(SA(1, 0), cA + TSTR); STAGE(SB(1, 1), cB + (TSTR + 16384));
        WAIT_V(6); BAR;
        LDB(B0, 0, 0);
        WAIT_L(0);
        const int hf = tile_hf(ntile - 1);
        const int nfull = ntile - (hf ? 1 : 0);
        for (int it = 0; it < nfull; ++it) {
            const bool has_next = it + 1 < ntile;
            const int nci = has_next ? tile_ci(it + 1) : ci, nrb = has_next ? tile_rb(it + 1) : rb, ncb = gcb(rank, nci);
            const int nhf = it + 1 == nfull ? hf : 0;
            int pp = 0;
            while (pp < 32) {
                unsigned a2, b2; int run;
                if (pp < 31) {
                    if (2 * pp + 3 >= lim) {
                        const int seg = (2 * pp + 2) / 7, nr = peek();
                        lim = nr > seg ? 1 + 7 * nr : ready(rb, cb, seg);
                    }
                    pre = false;
                    run = ((lim - 2) >> 1) - pp;
                    if (run > 1) --run;
                    else if (lim < 64) prepoll(rb, cb);
                    else if (has_next) prepoll(nrb, ncb);
                    a2 = cA + 2 * TSTR; b2 = cB + 2 * TSTR;
                } else {
                    if (has_next) { const int seg = nhf ? 8 : 0, nr = peek(); lim = nr > seg ? 1 + 7 * nr : ready(nrb, ncb, seg); }
                    pre = false;
                    run = 1;
                    a2 = (unsigned)(16 + ncb) * BSTR; b2 = (unsigned)nrb * BSTR + (nhf == 2 ? 16384u : 0u);
                }
                pp += run;
                do { KSTEP(1) } while (--run > 0);
            }
            epilogue(IC<2>{}, rb * 256);
#pragma unroll
            for (int a = 0; a < 2; ++a)
#pragma unroll
                for (int c = 0; c < 2; ++c)
#pragma unroll
                    for (int m = 0; m < 4; ++m)
#pragma unroll
                        for (int n = 0; n < 2; ++n) acc[a][c][m][n] = (f32x4){0.f, 0.f, 0.f, 0.f};
            ci = nci; rb = nrb; cb = ncb;
        }
        if (LEXC > 0 && hf) {
            const unsigned a0 = cA, b0 = cB;
            unsigned a2 = cA + 2 * TSTR, b2 = cB + 2 * TSTR;
            for (int pp = 0; pp < 32; ++pp) {
                if (pp == 31) { a2 = a0; b2 = b0; }
                KSTEP(0)
            }
            epilogue(IC<1>{}, rb * 256 + (hf == 2 ? 128 : 0));
        }
        WAIT_V(0);
        if (wr == 0) BAR;
    }
}

}

extern "C" void kernel_launch(int rank, int nranks, ncclComm_t comm,
                              hipStream_t stream,
                              void* const* d_in, const int* in_sizes, int n_in,
                              void* d_out, int out_size,
                              const int* in_dtypes, int out_dtype,
                              void* const* d_out_all,
                              void* const* d_ws_all, size_t ws_bytes) {
    (void)comm; (void)in_sizes; (void)in_dtypes; (void)out_dtype; (void)d_out_all; (void)out_size;
    if (nranks != NRANK || n_in < 2 || ws_bytes < WS_NEED || (unsigned)rank >= (unsigned)NRANK) {
        fprintf(stderr, "unexpected configuration\n");
        return;
    }
    HIP_CHECK(hipFuncSetAttribute((const void*)fused_gemm_allgather_silu, hipFuncAttributeMaxDynamicSharedMemorySize, LDS_BYTES));
    Params p;
    p.x = (const unsigned short*)d_in[0];
    p.w = (const unsigned short*)d_in[1];
    p.out = (unsigned short*)d_out;
    for (int i = 0; i < NRANK; ++i) p.ws[i] = (char*)d_ws_all[i];
    p.rank = rank;
    hipLaunchKernelGGL(fused_gemm_allgather_silu, dim3(NWG), dim3(512), LDS_BYTES, stream, p);
    HIP_CHECK(hipGetLastError());
}
```
